```python
import numpy as np

kernel_name = "gpu_gemm_ar_m4096_k4096_n16384_bf16_none"


ABSMAX_RTOL = 0.008

COMPUTE_DTYPE = "bfloat16"
EPILOGUE = "none"
IN_QUANT = None
OUT_QUANT = None
IN_DTYPES = {"x": "bfloat16", "w_mat": "bfloat16"}
WORLD_SIZE = 8

def _to_bf16(x):
    u = np.ascontiguousarray(x, dtype=np.float32).view(np.uint32)
    return ((u + 0x7FFF + ((u >> 16) & 1)) & 0xFFFF0000).view(np.float32)

def setup_inputs():

    x = _to_bf16(np.random.randn(4096, 4096).astype(np.float32))
    w_mat = _to_bf16((np.random.randn(4096, 16384) / np.sqrt(4096)).astype(np.float32))
    return {"x": x, "w_mat": w_mat}

def _epilogue(y):
    return y

def reference(x, w_mat):
    y = (x @ w_mat).astype(np.float32)
    return _epilogue(y)
```

```cpp
#include <hip/hip_runtime.h>
#include <rccl/rccl.h>
#include <cstdio>
#include <cstdint>

#define HIP_CHECK(expr) do { hipError_t _e = (expr); if (_e != hipSuccess) { \
    fprintf(stderr, "HIP error %s at %s:%d\n", hipGetErrorString(_e), __FILE__, __LINE__); \
    return; } } while (0)

namespace {
#define LAS __attribute__((address_space(3)))
#define GAS __attribute__((address_space(1)))
#define CAS __attribute__((address_space(4)))
typedef short bf16x8 __attribute__((ext_vector_type(8)));
typedef float f32x4 __attribute__((ext_vector_type(4)));
typedef unsigned u32x4 __attribute__((ext_vector_type(4)));
typedef unsigned long long u64;

constexpr int NR = 8;
constexpr int MM = 4096, NN = 16384, KS = 512;
constexpr unsigned IMG = 16384;
constexpr unsigned KTB = 2 * IMG;
constexpr unsigned CHUNK = 8 * KTB;
constexpr unsigned PANEL = NR * CHUNK;
constexpr int NXP = MM / 256, NWP = NN / 256;
constexpr int WPR = NWP / NR;
constexpr unsigned OFF_X = 1u << 20;
constexpr unsigned OFF_W = OFF_X + NXP * PANEL;
constexpr unsigned WS_USED = OFF_W + NWP * PANEL;
constexpr unsigned OUT_BYTES = (unsigned)MM * NN * 2;
constexpr unsigned OUT_PITCH = NN * 2;
constexpr int C_CALL = 0, C_FIN = 32, C_TMO = 64, C_DONE = 128  ;
constexpr int C_STG = 1024  ;
constexpr int C_YF = 8192  ;
constexpr int C_ARR = 16384  ;
constexpr int NCHUNK = NXP + NWP;
constexpr int GRID = 256, NPULL = 56, NWORK = GRID - NPULL;
constexpr int NSEQ = NXP + WPR;
constexpr int NTILE = NXP * WPR;
constexpr int NJOB = NCHUNK * 4;
constexpr unsigned LDS_BYTES = 8 * IMG + 256;
constexpr u64 DEADLINE = 40000000ull;
constexpr int AUX_SYS = 17;
constexpr int AUX_AGT = 16;

struct Params {
    const unsigned short* x;
    const unsigned short* w;
    char* ws[NR];
    char* out[NR];
    int rank;
};

__device__ __forceinline__ int lds_byte(int r, int c) { const int st = (r >> 4) * 2 + (c >> 5), rr = r & 15, cc = c & 31, ob = rr * 64 + cc * 2; return st * 1024 + (ob ^ (((ob >> 9) & 1) << 5)); }
__device__ __forceinline__ void stage_rc(int b, int& R, int& C) { const int st = b / 1024, sb = b % 1024, swz = sb ^ (((sb >> 9) & 1) << 5); R = (st >> 1) * 16 + swz / 64; C = (st & 1) * 32 + (swz % 64) / 2; }

__device__ __forceinline__ int seq_chunk(int s, int dst_rank) { const int g = s / 3, r = s % 3; return r == 0 ? NXP + dst_rank * WPR + g : 2 * g + r - 1; }
__device__ __forceinline__ void tile_of(int o, int& ti, int& tj) {
    int g = 0;
#pragma unroll
    for (int q = 1; q < 8; ++q) g += (o >= 2 * q * q) ? 1 : 0;
    const int r = o - 2 * g * g;
    if (r < 2 * g) { ti = r; tj = g; }
    else if (r < 3 * g + 1) { ti = 2 * g; tj = r - 2 * g; }
    else { ti = 2 * g + 1; tj = r - 3 * g - 1; }
}
__device__ __forceinline__ unsigned chunk_off(int cid, int src) { return (cid < NXP ? OFF_X + (unsigned)cid * PANEL : OFF_W + (unsigned)(cid - NXP) * PANEL) + (unsigned)src * CHUNK; }

__device__ __forceinline__ __amdgpu_buffer_rsrc_t mk_rsrc(const void* p, unsigned bytes) {
    const u64 a = (u64)p;
    const unsigned lo = __builtin_amdgcn_readfirstlane((unsigned)a), hi = __builtin_amdgcn_readfirstlane((unsigned)(a >> 32));
    return __builtin_amdgcn_make_buffer_rsrc((void*)(((u64)hi << 32) | lo), 0, bytes, 0x00020000);
}
__device__ __forceinline__ unsigned ld_sys(const unsigned* p) { return __hip_atomic_load(p, __ATOMIC_RELAXED, __HIP_MEMORY_SCOPE_SYSTEM); }
__device__ __forceinline__ unsigned ld_agt(const unsigned* p) { return __hip_atomic_load(p, __ATOMIC_RELAXED, __HIP_MEMORY_SCOPE_AGENT); }
__device__ __forceinline__ void st_sys(unsigned* p, unsigned v) { __hip_atomic_store(p, v, __ATOMIC_RELAXED, __HIP_MEMORY_SCOPE_SYSTEM); }
__device__ __forceinline__ void st_agt(unsigned* p, unsigned v) { __hip_atomic_store(p, v, __ATOMIC_RELAXED, __HIP_MEMORY_SCOPE_AGENT); }
#define WAIT_V(n) asm volatile("s_waitcnt vmcnt(" #n ")" ::: "memory")
#define WAIT_L(n) asm volatile("s_waitcnt lgkmcnt(" #n ")" ::: "memory")
#define BAR __builtin_amdgcn_s_barrier()
#define SCHED __builtin_amdgcn_sched_barrier(0)

__device__ __forceinline__ void signal_ranks(const Params& P, unsigned mask, int word, unsigned val, int lane) {
    u64 a = 0;
#pragma unroll
    for (int p = 0; p < NR; ++p) { u64 t = (u64)P.ws[p]; asm volatile("" : "+s"(t)); a = (lane == p) ? t : a; }
    if (lane < NR && ((mask >> lane) & 1u)) __hip_atomic_store((GAS unsigned*)a + word, val, __ATOMIC_RELAXED, __HIP_MEMORY_SCOPE_SYSTEM);
}
__device__ __forceinline__ bool expired(unsigned* ctl, unsigned call, u64 t0, int lane) {
    const bool dead = (unsigned)__builtin_amdgcn_readfirstlane(ld_agt(ctl + C_TMO)) == call || __builtin_amdgcn_s_memrealtime() - t0 > DEADLINE;
    if (dead && lane == 0) st_agt(ctl + C_TMO, call);
    return dead;
}

__device__ __forceinline__ void puller(const Params& P, int b, unsigned call) {
    const int rank = P.rank;
    const int wid = __builtin_amdgcn_readfirstlane(threadIdx.x >> 6), lane = threadIdx.x & 63;
    const int j = b & 7;
    const unsigned p = (unsigned)(rank + 1 + (b >> 3)) & 7u;
    unsigned* ctl = (unsigned*)P.ws[rank];
    const __amdgpu_buffer_rsrc_t s_ws = mk_rsrc(P.ws[p], WS_USED), d_ws = mk_rsrc(P.ws[rank], WS_USED);
    const __amdgpu_buffer_rsrc_t s_out = mk_rsrc(P.out[p], OUT_BYTES), d_out = mk_rsrc(P.out[rank], OUT_BYTES);
    bool dead = false;
    int pend = -1;
    {
        const unsigned* fl = ctl + C_STG + p * 512 + seq_chunk(lane < NSEQ ? lane : 0, rank) * 4 + (j >> 1);
        u64 rdy = 0;
        for (int s = 0; s < NSEQ; ++s) {
            const int cid = seq_chunk(s, rank);
            if (!((rdy >> s) & 1) && !dead) {
                const u64 t0 = __builtin_amdgcn_s_memrealtime();
                for (;;) {
                    const unsigned f = ld_sys(fl);
                    WAIT_V(0);
                    rdy = __ballot(f >= call);
                    if (pend >= 0) { if (lane == 0) __hip_atomic_fetch_add(ctl + C_ARR + pend * 8 + p, 1u, __ATOMIC_RELAXED, __HIP_MEMORY_SCOPE_AGENT); pend = -1; }
                    if ((rdy >> s) & 1) break;
                    if (expired(ctl, call, t0, lane)) { dead = true; break; }
                    __builtin_amdgcn_s_sleep(1);
                }
            }
            const unsigned off = chunk_off(cid, (int)p) + (unsigned)j * KTB + (unsigned)wid * 4096u + (unsigned)lane * 16u;
            if (!dead) {
#pragma unroll
                for (int q = 0; q < 4; ++q) {
                    unsigned f = 0;
                    if (q == 0) f = ld_sys(fl);
                    const u32x4 v = __builtin_amdgcn_raw_buffer_load_b128(s_ws, off + q * 1024u, 0, AUX_SYS);
                    WAIT_V(0);
                    __builtin_amdgcn_raw_buffer_store_b128(v, d_ws, off + q * 1024u, 0, AUX_AGT);
                    if (q == 0) {
                        rdy = __ballot(f >= call);
                        if (pend >= 0) { if (lane == 0) __hip_atomic_fetch_add(ctl + C_ARR + pend * 8 + p, 1u, __ATOMIC_RELAXED, __HIP_MEMORY_SCOPE_AGENT); pend = -1; }
                    }
                }
            } else if (pend >= 0) {
                WAIT_V(0);
                if (lane == 0) __hip_atomic_fetch_add(ctl + C_ARR + pend * 8 + p, 1u, __ATOMIC_RELAXED, __HIP_MEMORY_SCOPE_AGENT);
            }
            pend = cid;
        }
    }
    {
        const unsigned* fl = ctl + C_YF + p * 128 + lane;
        int t0i, t0j, t1i, t1j;
        tile_of(lane, t0i, t0j); tile_of(64 + lane, t1i, t1j);
        const unsigned tab = (unsigned)(t0i | (t0j << 4) | (t1i << 8) | (t1j << 12));
        const unsigned voff = (unsigned)(4 * (8 * j + wid) + (lane >> 5)) * OUT_PITCH + p * (WPR * 512u) + (unsigned)(lane & 31) * 16u;
        u64 r0 = 0, r1 = 0, d0 = 0, d1 = 0;
        for (int n = 0; n < NTILE; ++n) {
            if (dead) { r0 = ~0ull; r1 = ~0ull; }
            if (!((r0 & ~d0) | (r1 & ~d1))) {
                const u64 t0 = __builtin_amdgcn_s_memrealtime();
                for (;;) {
                    const unsigned f0 = ld_sys(fl), f1 = ld_sys(fl + 64);
                    WAIT_V(0);
                    r0 = __ballot(f0 >= call); r1 = __ballot(f1 >= call);
                    if (pend >= 0) { if (lane == 0) __hip_atomic_fetch_add(ctl + C_ARR + pend * 8 + p, 1u, __ATOMIC_RELAXED, __HIP_MEMORY_SCOPE_AGENT); pend = -1; }
                    if ((r0 & ~d0) | (r1 & ~d1)) break;
                    if (expired(ctl, call, t0, lane)) { dead = true; r0 = ~0ull; r1 = ~0ull; break; }
                    __builtin_amdgcn_s_sleep(1);
                }
            }
            const u64 a0 = r0 & ~d0, a1 = r1 & ~d1;
            unsigned e;
            if (a0) { const int o = __builtin_ctzll(a0); d0 |= 1ull << o; e = (unsigned)__builtin_amdgcn_readlane((int)tab, o) & 255u; }
            else    { const int o = __builtin_ctzll(a1); d1 |= 1ull << o; e = ((unsigned)__builtin_amdgcn_readlane((int)tab, o) >> 8) & 255u; }
            const unsigned soff = (e & 15u) * (256u * OUT_PITCH) + (e >> 4) * 512u;
            if (!dead) {
                const unsigned f0 = ld_sys(fl), f1 = ld_sys(fl + 64);
                const u32x4 v0 = __builtin_amdgcn_raw_buffer_load_b128(s_out, voff + soff, 0, AUX_SYS);
                WAIT_V(0);
                __builtin_amdgcn_raw_buffer_store_b128(v0, d_out, voff + soff, 0, AUX_AGT);
                r0 = __ballot(f0 >= call); r1 = __ballot(f1 >= call);
                if (pend >= 0) { if (lane == 0) __hip_atomic_fetch_add(ctl + C_ARR + pend * 8 + p, 1u, __ATOMIC_RELAXED, __HIP_MEMORY_SCOPE_AGENT); pend = -1; }
                const u32x4 v1 = __builtin_amdgcn_raw_buffer_load_b128(s_out, voff + soff + 2 * OUT_PITCH, 0, AUX_SYS);
                WAIT_V(0);
                __builtin_amdgcn_raw_buffer_store_b128(v1, d_out, voff + soff + 2 * OUT_PITCH, 0, AUX_AGT);
            } else {
                if (pend >= 0) { WAIT_V(0); if (lane == 0) __hip_atomic_fetch_add(ctl + C_ARR + pend * 8 + p, 1u, __ATOMIC_RELAXED, __HIP_MEMORY_SCOPE_AGENT); pend = -1; }
                const u32x4 nanv = {0x7fc07fc0u, 0x7fc07fc0u, 0x7fc07fc0u, 0x7fc07fc0u};
                __builtin_amdgcn_raw_buffer_store_b128(nanv, d_out, voff + soff, 0, AUX_AGT);
                __builtin_amdgcn_raw_buffer_store_b128(nanv, d_out, voff + soff + 2 * OUT_PITCH, 0, AUX_AGT);
            }
        }
    }
}

__device__ __forceinline__ void job_of(int idx, int& cid, int& q) {
    const int g = idx / 40, r = idx % 40;
    if (r < 32) { cid = NXP + (r >> 2) * WPR + g; q = r & 3; }
    else { cid = 2 * g + ((r - 32) >> 2); q = r & 3; }
}
__device__ __forceinline__ void stager(const Params& P, LAS unsigned char* lds, int g, unsigned call) {
    const int rank = P.rank, tid = threadIdx.x, lane = tid & 63;
    const __amdgpu_buffer_rsrc_t d_ws = mk_rsrc(P.ws[rank], WS_USED);
    const __amdgpu_buffer_rsrc_t s_x = mk_rsrc(P.x, MM * KS * 2), s_w = mk_rsrc(P.w, KS * NN * 2);
    for (int idx = g; idx < NJOB; idx += NWORK) {
        int cid, q; job_of(idx, cid, q);
        const unsigned dst = chunk_off(cid, rank) + (unsigned)q * (2 * KTB) + (unsigned)tid * 16u;
        if (cid < NXP) {
            u32x4 v[8];
#pragma unroll
            for (int u = 0; u < 8; ++u) {
                int R, C; stage_rc((u & 1) * 8192 + tid * 16, R, C);
                v[u] = __builtin_amdgcn_raw_buffer_load_b128(s_x, (unsigned)(R * KS + C) * 2u, (unsigned)((256 * cid + 128 * ((u >> 1) & 1)) * KS + 128 * q + 64 * (u >> 2)) * 2u, 0);
            }
#pragma unroll
            for (int u = 0; u < 8; ++u) __builtin_amdgcn_raw_buffer_store_b128(v[u], d_ws, dst + u * 8192u, 0, AUX_SYS);
        } else {
            const int jg = cid - NXP, ng = tid & 31, kg = tid >> 5;
            u32x4 v[8];
#pragma unroll
            for (int r = 0; r < 8; ++r) v[r] = __builtin_amdgcn_raw_buffer_load_b128(s_w, (unsigned)(8 * kg * NN + 8 * ng) * 2u, (unsigned)((128 * q + r) * NN + 256 * jg) * 2u, 0);
            const int lbase = (kg >> 3) * (int)KTB + (ng >> 4) * (int)IMG;
#pragma unroll
            for (int jj = 0; jj < 8; ++jj) {
                u32x4 o;
#pragma unroll
                for (int d = 0; d < 4; ++d)
                    o[d] = (jj & 1) ? __builtin_amdgcn_perm(v[2 * d + 1][jj >> 1], v[2 * d][jj >> 1], 0x07060302u)
                                    : __builtin_amdgcn_perm(v[2 * d + 1][jj >> 1], v[2 * d][jj >> 1], 0x05040100u);
                const int R = 32 * ((ng & 15) >> 2) + 16 * (jj >> 2) + 4 * (ng & 3) + (jj & 3);
                *(LAS u32x4*)(lds + lbase + lds_byte(R, (kg & 7) * 8)) = o;
            }
            __syncthreads();
#pragma unroll
            for (int u = 0; u < 8; ++u) v[u] = *(const LAS u32x4*)(lds + u * 8192 + tid * 16);
#pragma unroll
            for (int u = 0; u < 8; ++u) __builtin_amdgcn_raw_buffer_store_b128(v[u], d_ws, dst + u * 8192u, 0, AUX_SYS);
        }
        WAIT_V(0);
        __syncthreads();
        if (tid < 64) signal_ranks(P, cid < NXP ? 0xffu : (1u << ((cid - NXP) / WPR)), C_STG + rank * 512 + cid * 4 + q, call, lane);
    }
}

__device__ __forceinline__ unsigned cvt_pk_bf16(float lo, float hi) { unsigned r; asm volatile("v_cvt_pk_bf16_f32 %0, %1, %2" : "=v"(r) : "v"(lo), "v"(hi)); return r; }

__device__ __forceinline__ void wait_words(unsigned* ctl, const unsigned* word, unsigned target, bool active, unsigned call, int wid, int lane) {
    if (wid == 0) {
        const u64 t0 = __builtin_amdgcn_s_memrealtime();
        for (;;) {
            const unsigned f = active ? ld_sys(word) : target;
            if (__all(f >= target)) break;
            if (expired(ctl, call, t0, lane)) break;
            __builtin_amdgcn_s_sleep(2);
        }
        __builtin_amdgcn_fence(__ATOMIC_ACQUIRE, "agent");
        WAIT_V(0);
    }
    asm volatile("" ::: "memory"); BAR; asm volatile("" ::: "memory");
}

__device__ __forceinline__ void gemm_tile(const Params& P, LAS unsigned char* lds, int o, unsigned call) {
    const int rank = P.rank;
    const int tid = threadIdx.x, wid = __builtin_amdgcn_readfirstlane(tid >> 6), lane = tid & 63, wr = wid >> 2, wc = wid & 3, fr = lane & 15, fq = lane >> 4;
    unsigned* ctl = (unsigned*)P.ws[rank];
    int ti, tj; tile_of(o, ti, tj);
    const int jg = rank * WPR + tj;
    const char* cA = P.ws[rank] + OFF_X + (size_t)ti * PANEL + tid * 16;
    const char* cB = P.ws[rank] + OFF_W + (size_t)jg * PANEL + tid * 16;
    const unsigned ldsw = (unsigned)wid * 1024u;
    const int aoff = lds_byte(wr * 64 + fr, fq * 8), boff = lds_byte(wc * 32 + fr, fq * 8);
    const int kt0 = rank * 8;
#define KOFF(t) ((size_t)((unsigned)((kt0 + (t)) & 63) * KTB))
#define SA(b, h) (((b) * 2 + (h)) * IMG)
#define SB(b, h) ((4 + (b) * 2 + (h)) * IMG)
#define STAGE(bufoff, gbase) do { _Pragma("unroll") for (int _i = 0; _i < 2; ++_i) \
        __builtin_amdgcn_global_load_lds((const unsigned*)((gbase) + _i * 8192), (LAS unsigned*)(lds + (bufoff) + ldsw + _i * 8192), 16, 0, 0); } while (0)
#define LDA(dst, b, h) do { _Pragma("unroll") for (int m = 0; m < 4; ++m) _Pragma("unroll") for (int k = 0; k < 2; ++k) dst[m][k] = *(const LAS bf16x8*)(lds + SA(b, h) + aoff + m * 2048 + k * 1024); } while (0)
#define LDB(dst, b, h) do { _Pragma("unroll") for (int n = 0; n < 2; ++n) _Pragma("unroll") for (int k = 0; k < 2; ++k) dst[n][k] = *(const LAS bf16x8*)(lds + SB(b, h) + boff + n * 2048 + k * 1024); } while (0)
#define MMA(ai, bj, At, Bt) do { __builtin_amdgcn_s_setprio(1); _Pragma("unroll") for (int m = 0; m < 4; ++m) _Pragma("unroll") for (int n = 0; n < 2; ++n) _Pragma("unroll") for (int k = 0; k < 2; ++k) \
        acc[ai][bj][m][n] = __builtin_amdgcn_mfma_f32_16x16x32_bf16(Bt[n][k], At[m][k], acc[ai][bj][m][n], 0, 0, 0); __builtin_amdgcn_s_setprio(0); } while (0)
    f32x4 acc[2][2][4][2];
#pragma unroll
    for (int a = 0; a < 2; ++a)
#pragma unroll
        for (int b = 0; b < 2; ++b)
#pragma unroll
            for (int m = 0; m < 4; ++m)
#pragma unroll
                for (int n = 0; n < 2; ++n) acc[a][b][m][n] = (f32x4){0.f, 0.f, 0.f, 0.f};
    bf16x8 At[4][2], B0[2][2], B1[2][2];
    wait_words(ctl, ctl + C_STG + rank * 512 + ((lane & 4) ? NXP + jg : ti) * 4 + (lane & 3), call, lane < 8, call, wid, lane);
    STAGE(SB(0, 0), cB + KOFF(0)); STAGE(SA(0, 0), cA + KOFF(0)); STAGE(SB(0, 1), cB + KOFF(0) + IMG); STAGE(SA(0, 1), cA + KOFF(0) + IMG);
    if (wr == 1) BAR;
    WAIT_V(4); BAR;
    STAGE(SB(1, 0), cB + KOFF(1)); STAGE(SA(1, 0), cA + KOFF(1)); STAGE(SB(1, 1), cB + KOFF(1) + IMG);
    WAIT_V(6); BAR;
    for (int t = 0; t < 64; t += 2) {
        const char* a1 = cA + KOFF(t + 1);
        const char* a2 = cA + KOFF(t + 2); const char* b2 = cB + KOFF(t + 2);
        const char* a3 = cA + KOFF(t + 3); const char* b3 = cB + KOFF(t + 3);
        if (t == 6)
            wait_words(ctl, ctl + C_ARR + ((lane & 8) ? NXP + jg : ti) * 8 + (lane & 7), 64u * call, lane < 16 && (lane & 7) != rank, call, wid, lane);
        LDB(B0, 0, 0); SCHED; LDA(At, 0, 0); STAGE(SA(1, 1), a1 + IMG);
        WAIT_L(8); BAR; WAIT_L(0); MMA(0, 0, At, B0); BAR; SCHED;
        LDB(B1, 0, 1); STAGE(SB(0, 0), b2);
        BAR; WAIT_L(0); MMA(0, 1, At, B1); BAR;
        LDA(At, 0, 1); STAGE(SA(0, 0), a2);
        BAR; WAIT_L(0); MMA(1, 0, At, B0); BAR; SCHED;
        STAGE(SB(0, 1), b2 + IMG);
        WAIT_V(6); BAR; MMA(1, 1, At, B1); BAR;
        LDB(B0, 1, 0); SCHED; LDA(At, 1, 0); STAGE(SA(0, 1), a2 + IMG);
        WAIT_L(8); BAR; WAIT_L(0); MMA(0, 0, At, B0); BAR; SCHED;
        LDB(B1, 1, 1); STAGE(SB(1, 0), b3);
        BAR; WAIT_L(0); MMA(0, 1, At, B1); BAR;
        LDA(At, 1, 1); STAGE(SA(1, 0), a3);
        BAR; WAIT_L(0); MMA(1, 0, At, B0); BAR; SCHED;
        STAGE(SB(1, 1), b3 + IMG);
        WAIT_V(6); BAR; MMA(1, 1, At, B1); BAR;
    }
    WAIT_V(0);
    if (wr == 0) BAR;
    const bool bad = (unsigned)__builtin_amdgcn_readfirstlane(ld_agt(ctl + C_TMO)) == call;
    const __amdgpu_buffer_rsrc_t d_out = mk_rsrc(P.out[rank], OUT_BYTES);
    const unsigned obase = (unsigned)(256 * ti + wr * 64 + fr) * OUT_PITCH + (unsigned)(256 * jg + wc * 32 + 8 * fq) * 2u;
#pragma unroll
    for (int ai = 0; ai < 2; ++ai)
#pragma unroll
        for (int m = 0; m < 4; ++m)
#pragma unroll
            for (int bj = 0; bj < 2; ++bj) {
                const f32x4 v0 = acc[ai][bj][m][0], v1 = acc[ai][bj][m][1];
                u32x4 wv; wv.x = cvt_pk_bf16(v0[0], v0[1]); wv.y = cvt_pk_bf16(v0[2], v0[3]); wv.z = cvt_pk_bf16(v1[0], v1[1]); wv.w = cvt_pk_bf16(v1[2], v1[3]);
                if (bad) wv = (u32x4){0x7fc07fc0u, 0x7fc07fc0u, 0x7fc07fc0u, 0x7fc07fc0u};
                __builtin_amdgcn_raw_buffer_store_b128(wv, d_out, obase + (unsigned)(ai * 128 + m * 16) * OUT_PITCH + bj * 256u, 0, AUX_SYS);
            }
    WAIT_V(0);
    BAR;
    if (wid == 0) signal_ranks(P, 0xffu & ~(1u << rank), C_YF + rank * 128 + o, call, lane);
#undef KOFF
#undef SA
#undef SB
#undef STAGE
#undef LDA
#undef LDB
#undef MMA
}

__global__ __launch_bounds__(512, 2) void fused_gemm_ar(Params P) {
    extern __shared__ __attribute__((aligned(16))) unsigned char shm[];
    LAS unsigned char* lds = (LAS unsigned char*)shm;
    const int b = blockIdx.x, rank = P.rank;
    unsigned* ctl = (unsigned*)P.ws[rank];
    const unsigned call = *(const CAS unsigned*)(u64)(ctl + C_CALL) + 1u;
    if (b < NPULL) {
        puller(P, b, call);
    } else {
        const int g = b - NPULL;
        stager(P, lds, g, call);
        if (g < NTILE) gemm_tile(P, lds, g, call);
    }
    WAIT_V(0);
    __syncthreads();
    if (threadIdx.x < 64) {
        const int lane = threadIdx.x;
            unsigned ticket = 0;
        if (lane == 0) ticket = __hip_atomic_fetch_add(ctl + C_FIN, 1u, __ATOMIC_RELAXED, __HIP_MEMORY_SCOPE_AGENT);
        ticket = (unsigned)__builtin_amdgcn_readfirstlane(ticket);
        if (ticket % GRID == GRID - 1) {
            signal_ranks(P, 0xffu & ~(1u << rank), C_DONE + 32 * rank, call, lane);
            const u64 t0 = __builtin_amdgcn_s_memrealtime();
            for (;;) {
                const unsigned f = (lane < NR && lane != rank) ? ld_sys(ctl + C_DONE + 32 * lane) : call;
                if (__all(f >= call)) break;
                if (expired(ctl, call, t0, lane)) { if (lane == 0) st_sys((unsigned*)P.out[rank], 0x7fc07fc0u); break; }
                __builtin_amdgcn_s_sleep(1);
            }
            if (lane == 0) st_agt(ctl + C_CALL, call);
        }
    }
}
}

extern "C" void kernel_launch(int rank, int nranks, ncclComm_t comm,
                              hipStream_t stream,
                              void* const* d_in, const int* in_sizes, int n_in,
                              void* d_out, int out_size,
                              const int* in_dtypes, int out_dtype,
                              void* const* d_out_all,
                              void* const* d_ws_all, size_t ws_bytes) {
    if (nranks != NR || (unsigned)rank >= (unsigned)NR || ws_bytes < WS_USED || n_in < 2 ||
        in_sizes[0] != MM * KS || in_sizes[1] != KS * NN || out_size != MM * NN) {
        fprintf(stderr, "kernel_launch: unexpected problem shape\n");
        return;
    }
    static unsigned attr_done = 0;
    int dev = 0; HIP_CHECK(hipGetDevice(&dev));
    if (!((attr_done >> (dev & 31)) & 1u)) {
        HIP_CHECK(hipFuncSetAttribute((const void*)fused_gemm_ar, hipFuncAttributeMaxDynamicSharedMemorySize, (int)LDS_BYTES));
        attr_done |= 1u << (dev & 31);
    }
    Params P;
    P.x = (const unsigned short*)d_in[0];
    P.w = (const unsigned short*)d_in[1];
    for (int p = 0; p < NR; ++p) { P.ws[p] = (char*)d_ws_all[p]; P.out[p] = (char*)d_out_all[p]; }
    P.rank = rank;
    hipLaunchKernelGGL(fused_gemm_ar, dim3(GRID), dim3(512), LDS_BYTES, stream, P);
    HIP_CHECK(hipGetLastError());
}
```

```cpp
#include <hip/hip_runtime.h>
#include <rccl/rccl.h>
#include <cstdio>
#include <cstdint>

#define HIP_CHECK(expr) do { hipError_t _e = (expr); if (_e != hipSuccess) { \
    fprintf(stderr, "HIP error %s at %s:%d\n", hipGetErrorString(_e), __FILE__, __LINE__); \
    return; } } while (0)

#define EREP 4
#define PUSH0 2
#define RTAIL 6
#define SPLIT 1
#define KTICKS 100

namespace {
#define LAS __attribute__((address_space(3)))
#define GAS __attribute__((address_space(1)))
#define CAS __attribute__((address_space(4)))
typedef short bf16x8 __attribute__((ext_vector_type(8)));
typedef float f32x4 __attribute__((ext_vector_type(4)));
typedef unsigned u32x4 __attribute__((ext_vector_type(4)));
typedef unsigned u32x2 __attribute__((ext_vector_type(2)));
typedef unsigned long long u64;

constexpr int NR = 8;
constexpr int MM = 4096, NN = 16384, KS = 512;
constexpr unsigned IMG = 16384;
constexpr unsigned KTB = 2 * IMG;
constexpr unsigned CHUNK = 8 * KTB;
constexpr unsigned PANEL = NR * CHUNK;
constexpr int NXP = MM / 256, NWP = NN / 256;
constexpr int WPR = NWP / NR;
constexpr int NSH = WPR - EREP;
constexpr int NW = NSH + NR * EREP;
constexpr unsigned OFF_X = 1u << 20;
constexpr unsigned OFF_W = OFF_X + NXP * PANEL;
constexpr unsigned PAR_STRIDE = (NXP + NWP) * PANEL;
constexpr unsigned OFF_SLAB = OFF_X + 2 * PAR_STRIDE;
constexpr unsigned SLAB = 256 * 256 * 4;
constexpr unsigned WS_USED = OFF_SLAB + 8 * RTAIL * SLAB;
constexpr unsigned OUT_BYTES = (unsigned)MM * NN * 2;
constexpr unsigned OUT_PITCH = NN * 2;
constexpr int NCHUNK = NXP + NWP;
constexpr int NPUSH = PUSH0 == 1 ? 1 : 0;
constexpr int XPUSH = PUSH0 == 2 ? 1 : 0;
constexpr int NSEQ = NXP + NW - NPUSH - XPUSH;
constexpr int NYT = NXP * NSH;
constexpr int NIN_IT = NSEQ * 64;
constexpr int NTOT_IT = NIN_IT + NYT * 64;
constexpr int GRID = 256, NPULL = 56, NWORK = GRID - NPULL, WPG = NWORK / 8;
constexpr int NJOB = NCHUNK * 4;
constexpr int C_FIN = 32, C_TMO = 64, C_SDONE = 96;
constexpr int C_XP = 128  ;
constexpr int C_Q = 512  , C_GQ = 736, C_TQ = 768  ;
constexpr int C_STG = 1024  ;
constexpr int C_YF = 8192  ;
constexpr int C_ARR = 12288  ;
constexpr int C_SLAB = 13312  ;
constexpr int C_ACNT = 16384  ;
constexpr unsigned LDS_BYTES = 8 * IMG + 256;
constexpr u64 DEADLINE = 40000000ull;
constexpr int AUX_SYS = 17;
constexpr int AUX_AGT = 16;
static_assert(RTAIL <= 16 && NSEQ <= 64 && NYT <= 128 && (C_ACNT + 64 * 8 * 32) * 4 <= (512 << 10), "layout");

struct Params {
    const unsigned short* x;
    const unsigned short* w;
    char* ws[NR];
    char* out[NR];
    int rank;
};

constexpr int pos_x(int i) { return 2 * i + 1 - XPUSH; }
constexpr int pos_w(int b) { return b < NXP ? 2 * b + XPUSH : b + NXP; }
constexpr int tile_pos(int i, int b) { return pos_x(i) > pos_w(b) ? pos_x(i) : pos_w(b); }
constexpr int NSEQ_POS = (NW >= NXP) ? NXP + NW : 2 * NXP;
struct Tables {
    int tile[8][96];
    int ntile[8];
    int gtile[2 * 8 * 16];
    int ngt;
    int ytile[128];
    int seq[64];
};
constexpr Tables make_tables() {
    Tables T{};
    int ny = 0, ns = 0;
    for (int pos = 0; pos < NSEQ_POS; ++pos) {
        for (int i = XPUSH; i < NXP; ++i) if (pos_x(i) == pos) T.seq[ns++] = i;
        for (int b = NPUSH; b < NW; ++b) if (pos_w(b) == pos) T.seq[ns++] = -1 - b;
        for (int b = 0; b < NW; ++b)
            for (int i = 0; i < NXP; ++i)
                if (tile_pos(i, b) == pos) {
                    const int q = (i & 3) + 4 * (b & 1);
                    T.tile[q][T.ntile[q]++] = i | (b << 4) | (b < NSH ? ny << 12 : 0);
                    if (b < NSH) T.ytile[ny++] = i | (b << 4);
                }
    }
    int gt[8 * 16] = {}; int ng = 0;
    for (int k = RTAIL; k >= 1; --k)
        for (int q = 0; q < 8; ++q)
            if (T.ntile[q] - RTAIL >= WPG) gt[ng++] = T.tile[q][T.ntile[q] - k];
    for (int q = 0; q < 8; ++q)
        if (T.ntile[q] - RTAIL >= WPG) T.ntile[q] -= RTAIL;
    constexpr int LAG = 8;
    for (int k = 0; k < ng + LAG; ++k) {
        if (k < ng) T.gtile[T.ngt++] = SPLIT ? gt[k] | (1 << 20) | (k << 22) : gt[k];
        if (k >= LAG && SPLIT) T.gtile[T.ngt++] = gt[k - LAG] | (2 << 20) | ((k - LAG) << 22);
    }
    return T;
}
__constant__ const Tables TAB = make_tables();
constexpr int n_active() { const Tables T = make_tables(); int n = 0; for (int q = 0; q < 8; ++q) n += T.ntile[q] < WPG ? T.ntile[q] : WPG; return n; }
constexpr int NACTIVE = n_active();

__device__ __forceinline__ int lds_byte(int r, int c) { const int st = (r >> 4) * 2 + (c >> 5), rr = r & 15, cc = c & 31, ob = rr * 64 + cc * 2; return st * 1024 + (ob ^ (((ob >> 9) & 1) << 5)); }
__device__ __forceinline__ void stage_rc(int b, int& R, int& C) { const int st = b / 1024, sb = b % 1024, swz = sb ^ (((sb >> 9) & 1) << 5); R = (st >> 1) * 16 + swz / 64; C = (st & 1) * 32 + (swz % 64) / 2; }
__device__ __forceinline__ int wpanel(int b, int r) { const int m = b - NSH; return b < NSH ? r * WPR + b : (m & 7) * WPR + NSH + (m >> 3); }
__device__ __forceinline__ int seq_chunk(int s, int r) { const int v = TAB.seq[s]; return v >= 0 ? v : NXP + wpanel(-1 - v, r); }
__device__ __forceinline__ unsigned chunk_off(int cid, int src, unsigned call) { return OFF_X + (call & 1u) * PAR_STRIDE + (unsigned)cid * PANEL + (unsigned)src * CHUNK; }

__device__ __forceinline__ __amdgpu_buffer_rsrc_t mk_rsrc(const void* p, unsigned bytes) {
    const u64 a = (u64)p;
    const unsigned lo = __builtin_amdgcn_readfirstlane((unsigned)a), hi = __builtin_amdgcn_readfirstlane((unsigned)(a >> 32));
    return __builtin_amdgcn_make_buffer_rsrc((void*)(((u64)hi << 32) | lo), 0, bytes, 0x00020000);
}
__device__ __forceinline__ unsigned ld_sys(const unsigned* p) { return __hip_atomic_load(p, __ATOMIC_RELAXED, __HIP_MEMORY_SCOPE_SYSTEM); }
__device__ __forceinline__ unsigned ld_agt(const unsigned* p) { return __hip_atomic_load(p, __ATOMIC_RELAXED, __HIP_MEMORY_SCOPE_AGENT); }
__device__ __forceinline__ void st_sys(unsigned* p, unsigned v) { __hip_atomic_store(p, v, __ATOMIC_RELAXED, __HIP_MEMORY_SCOPE_SYSTEM); }
__device__ __forceinline__ void st_agt(unsigned* p, unsigned v) { __hip_atomic_store(p, v, __ATOMIC_RELAXED, __HIP_MEMORY_SCOPE_AGENT); }
#define RFL(x) __builtin_amdgcn_readfirstlane(x)
#define ST128(v, rsrc, voff, soff, aux) __builtin_amdgcn_raw_buffer_store_b128(v, rsrc, (voff) + (soff), 0, aux)
#define STAMP(slot) do { } while (0)
#define WAIT_V(n) asm volatile("s_waitcnt vmcnt(" #n ")" ::: "memory")
#define WAIT_L(n) asm volatile("s_waitcnt lgkmcnt(" #n ")" ::: "memory")
#define BAR __builtin_amdgcn_s_barrier()
#define SCHED __builtin_amdgcn_sched_barrier(0)
#define LANE0_ADD(rsrc, word, lane) (unsigned)__builtin_amdgcn_raw_ptr_buffer_atomic_add_i32(1, rsrc, (lane) == 0 ? (int)((word) * 4) : (int)0xfffffff0u, 0, 0)

__device__ __forceinline__ void signal_ranks(const Params& P, unsigned mask, int word, unsigned val, int lane) {
    const CAS Params* kp = (const CAS Params*)__builtin_amdgcn_kernarg_segment_ptr();
    asm volatile("" : "+s"(kp));
    u64 a = 0;
#pragma unroll
    for (int p = 0; p < NR; ++p) { u64 t = (u64)kp->ws[p]; asm volatile("" : "+s"(t)); a = (lane == p) ? t : a; }
    if (lane < NR && ((mask >> lane) & 1u)) __hip_atomic_store((GAS unsigned*)a + word, val, __ATOMIC_RELAXED, __HIP_MEMORY_SCOPE_SYSTEM);
}
__device__ __forceinline__ bool expired(unsigned* ctl, unsigned call, u64 t0, int lane) {
    const u64 dt = __builtin_amdgcn_s_memrealtime() - t0;
    if (dt < 100000ull) return false;
    const bool dead = dt > DEADLINE || (unsigned)RFL(ld_agt(ctl + C_TMO)) == call;
    if (dead && lane == 0) st_agt(ctl + C_TMO, call);
    return RFL((int)dead) != 0;
}

__device__ __forceinline__ void puller(const Params& P, LAS unsigned char* lds, int b, unsigned call) {
    const int rank = P.rank, lane = threadIdx.x & 63, link = b >> 3;
    const unsigned p = (unsigned)(rank + 1 + link) & 7u;
    unsigned* ctl = (unsigned*)P.ws[rank];
    const __amdgpu_buffer_rsrc_t s_ws = mk_rsrc(P.ws[p], WS_USED), d_ws = mk_rsrc(P.ws[rank], WS_USED);
    const __amdgpu_buffer_rsrc_t s_out = mk_rsrc(P.out[p], OUT_BYTES), d_out = mk_rsrc(P.out[rank], OUT_BYTES);
    const unsigned qbase = (call - 1u) * (unsigned)(NTOT_IT + 64);
    const int qword = C_Q + 32 * link;
    bool dead = false;
    bool xp_pend = false;
    LAS unsigned* xcnt = (LAS unsigned*)lds;
    if (XPUSH) {
        const int k = (b & 7) * 8 + RFL((int)(threadIdx.x >> 6));
        const __amdgpu_buffer_rsrc_t s_x = mk_rsrc(P.x, MM * KS * 2);
        u32x4 v[4];
#pragma unroll
        for (int q = 0; q < 4; ++q) {
            int R, C; stage_rc((k & 3) * 4096 + q * 1024 + lane * 16, R, C);
            v[q] = __builtin_amdgcn_raw_buffer_load_b128(s_x, (unsigned)(R * KS + C) * 2u, (unsigned)(128 * ((k >> 2) & 1) * KS + 64 * (k >> 3)) * 2u, 0);
        }
        if (threadIdx.x == 0) *xcnt = 0u;
        __syncthreads();
#pragma unroll
        for (int q = 0; q < 4; ++q) ST128(v[q], s_ws, (unsigned)lane * 16u, chunk_off(0, rank, call) + (unsigned)k * 4096u + q * 1024u, AUX_SYS);
        xp_pend = true;
    }
#define XP_DONE() do { if (XPUSH && xp_pend) { xp_pend = false; \
        if (lane == 0 && __hip_atomic_fetch_add(xcnt, 1u, __ATOMIC_RELAXED, __HIP_MEMORY_SCOPE_WORKGROUP) == 7u) \
            __hip_atomic_fetch_add((unsigned*)P.ws[p] + C_XP + 32 * rank, 1u, __ATOMIC_RELAXED, __HIP_MEMORY_SCOPE_SYSTEM); } } while (0)
    unsigned tk = LANE0_ADD(d_ws, qword, lane);
    const int mycid = seq_chunk(lane < NSEQ ? lane : 0, rank);
    const unsigned ytab = (unsigned)TAB.ytile[lane] | ((unsigned)TAB.ytile[64 + lane] << 8);
    unsigned nn = (unsigned)RFL(tk) - qbase;
    int pend = -1;
    {
        const unsigned foff = (unsigned)(C_STG + p * 512 + mycid * 4) * 4u;
        u64 rq0 = 0, rq1 = 0, rq2 = 0, rq3 = 0;
        while (nn < (unsigned)NIN_IT) {
            const unsigned n = nn;
            const int s = (int)(n >> 6), qq = (int)(n >> 4) & 3;
            bool rdy = (((qq == 0 ? rq0 : qq == 1 ? rq1 : qq == 2 ? rq2 : rq3) >> s) & 1) != 0;
            if (!rdy && !dead) {
                const u64 t0 = __builtin_amdgcn_s_memrealtime();
                for (;;) {
                    const u32x4 f = __builtin_amdgcn_raw_buffer_load_b128(d_ws, foff, 0, AUX_SYS);
                    rq0 = __ballot(f.x >= call); rq1 = __ballot(f.y >= call); rq2 = __ballot(f.z >= call); rq3 = __ballot(f.w >= call);
                    if (((qq == 0 ? rq0 : qq == 1 ? rq1 : qq == 2 ? rq2 : rq3) >> s) & 1) break;
                    if (xp_pend) { WAIT_V(0); XP_DONE(); }
                    if (pend >= 0) {
                        WAIT_V(0);
                        const unsigned c = (unsigned)RFL(LANE0_ADD(d_ws, C_ACNT + (pend * 8 + (int)p) * 32, lane));
                        if ((c & 63u) == 63u && lane == 0) st_agt(ctl + C_ARR + __builtin_amdgcn_readlane(mycid, pend) * 8 + p, call);
                        pend = -1;
                    }
                    if (expired(ctl, call, t0, lane)) { dead = true; break; }
                    __builtin_amdgcn_s_sleep(1);
                }
            }
            const int cid = __builtin_amdgcn_readlane(mycid, s);
            const unsigned soff = chunk_off(cid, (int)p, call) + (n & 63u) * 4096u, voff = (unsigned)lane * 16u;
            tk = LANE0_ADD(d_ws, qword, lane);
            u32x4 v = {0u, 0u, 0u, 0u};
            if (!dead) v = __builtin_amdgcn_raw_buffer_load_b128(s_ws, voff, soff, AUX_SYS);
            WAIT_V(0);
            nn = (unsigned)RFL(tk) - qbase;
            ST128(v, d_ws, voff, soff, AUX_AGT);
            XP_DONE();
            unsigned c = 0;
            if (pend >= 0) c = LANE0_ADD(d_ws, C_ACNT + (pend * 8 + (int)p) * 32, lane);
            if (!dead) v = __builtin_amdgcn_raw_buffer_load_b128(s_ws, voff, soff + 1024u, AUX_SYS);
            WAIT_V(0);
            ST128(v, d_ws, voff, soff + 1024u, AUX_AGT);
            if (pend >= 0 && ((unsigned)RFL(c) & 63u) == 63u && lane == 0) st_agt(ctl + C_ARR + __builtin_amdgcn_readlane(mycid, pend) * 8 + p, call);
            if (!dead) v = __builtin_amdgcn_raw_buffer_load_b128(s_ws, voff, soff + 2048u, AUX_SYS);
            ST128(v, d_ws, voff, soff + 2048u, AUX_AGT);
            if (!dead) v = __builtin_amdgcn_raw_buffer_load_b128(s_ws, voff, soff + 3072u, AUX_SYS);
            ST128(v, d_ws, voff, soff + 3072u, AUX_AGT);
            pend = s;
        }
        if (xp_pend) { WAIT_V(0); XP_DONE(); }
        if (pend >= 0) {
            WAIT_V(0);
            const unsigned c = (unsigned)RFL(LANE0_ADD(d_ws, C_ACNT + (pend * 8 + (int)p) * 32, lane));
            if ((c & 63u) == 63u && lane == 0) st_agt(ctl + C_ARR + __builtin_amdgcn_readlane(mycid, pend) * 8 + p, call);
        }
    }
    {
        const unsigned* fl = ctl + C_YF + p * 128 + lane;
        const unsigned voff = (unsigned)(lane >> 5) * OUT_PITCH + p * (WPR * 512u) + (unsigned)(lane & 31) * 16u;
        u64 r0 = 0, r1 = 0;
        while (nn < (unsigned)NTOT_IT) {
            const unsigned n = nn - (unsigned)NIN_IT;
            const int o = (int)(n >> 6);
            if (!dead && !(((o < 64 ? r0 : r1) >> (o & 63)) & 1)) {
                const u64 t0 = __builtin_amdgcn_s_memrealtime();
                for (;;) {
                    const unsigned f0 = ld_sys(fl), f1 = ld_sys(fl + 64);
                    r0 = __ballot(f0 >= call); r1 = __ballot(f1 >= call);
                    if (((o < 64 ? r0 : r1) >> (o & 63)) & 1) break;
                    if (expired(ctl, call, t0, lane)) { dead = true; break; }
                    __builtin_amdgcn_s_sleep(1);
                }
            }
            const unsigned e = ((unsigned)__builtin_amdgcn_readlane((int)ytab, o & 63) >> ((o >> 6) * 8)) & 255u;
            const unsigned soff = ((e & 15u) * 256u + (n & 63u) * 4u) * OUT_PITCH + (e >> 4) * 512u;
            if (!dead) {
                const u32x4 v0 = __builtin_amdgcn_raw_buffer_load_b128(s_out, voff, soff, AUX_SYS);
                tk = LANE0_ADD(d_ws, qword, lane);
                const int o2 = o + 3 < NYT ? o + 3 : NYT - 1;
                const bool refresh = !(((o2 < 64 ? r0 : r1) >> (o2 & 63)) & 1);
                unsigned f0 = 0, f1 = 0;
                if (refresh) { f0 = ld_sys(fl); f1 = ld_sys(fl + 64); }
                ST128(v0, d_out, voff, soff, AUX_AGT);
                const u32x4 v1 = __builtin_amdgcn_raw_buffer_load_b128(s_out, voff, soff + 2 * OUT_PITCH, AUX_SYS);
                nn = (unsigned)RFL(tk) - qbase;
                if (refresh) { r0 = __ballot(f0 >= call); r1 = __ballot(f1 >= call); }
                SCHED;
                ST128(v1, d_out, voff, soff + 2 * OUT_PITCH, AUX_AGT);
            } else {
                tk = LANE0_ADD(d_ws, qword, lane);
                nn = (unsigned)RFL(tk) - qbase;
                const u32x4 nanv = {0x7fc07fc0u, 0x7fc07fc0u, 0x7fc07fc0u, 0x7fc07fc0u};
                ST128(nanv, d_out, voff, soff, AUX_AGT);
                ST128(nanv, d_out, voff, soff + 2 * OUT_PITCH, AUX_AGT);
            }
        }
    }
}

__device__ __forceinline__ void job_of(int idx, int& cid, int& q) {
    q = idx & 3;
    int u = idx >> 2;
    if (u < 9 * NSH) { const int g = u / 9, r = u % 9; cid = r < 8 ? NXP + r * WPR + g : (g < NXP ? g : 0); return; }
    u -= 9 * NSH;
    constexpr int NXR = NXP - NSH, NPAIR = NXR < 8 * EREP ? NXR : 8 * EREP;
    if (u < 2 * NPAIR) { cid = (u & 1) ? NSH + (u >> 1) : NXP + wpanel(NSH + (u >> 1), 0); return; }
    u -= 2 * NPAIR;
    cid = NXR > NPAIR ? NSH + NPAIR + u : NXP + wpanel(NSH + NPAIR + u, 0);
}
__device__ __forceinline__ void stager(const Params& P, LAS unsigned char* lds, int g, unsigned call) {
    const int rank = P.rank, tid = threadIdx.x, lane = tid & 63;
    const __amdgpu_buffer_rsrc_t d_ws = mk_rsrc(P.ws[rank], WS_USED);
    const __amdgpu_buffer_rsrc_t s_x = mk_rsrc(P.x, MM * KS * 2), s_w = mk_rsrc(P.w, KS * NN * 2);
    for (int idx = g; idx < NJOB; idx += NWORK) {
        int cid, q; job_of(idx, cid, q);
        const unsigned dst = chunk_off(cid, rank, call) + (unsigned)q * (2 * KTB) + (unsigned)tid * 16u;
        if (cid < NXP) {
            u32x4 v[8];
#pragma unroll
            for (int u = 0; u < 8; ++u) {
                int R, C; stage_rc((u & 1) * 8192 + tid * 16, R, C);
                v[u] = __builtin_amdgcn_raw_buffer_load_b128(s_x, (unsigned)(R * KS + C) * 2u, (unsigned)((256 * cid + 128 * ((u >> 1) & 1)) * KS + 128 * q + 64 * (u >> 2)) * 2u, 0);
            }
#pragma unroll
            for (int u = 0; u < 8; ++u) ST128(v[u], d_ws, dst + u * 8192u, 0, AUX_SYS);
        } else {
            const int jg = cid - NXP, ng = tid & 31, kg = tid >> 5;
            u32x4 v[8];
#pragma unroll
            for (int r = 0; r < 8; ++r) v[r] = __builtin_amdgcn_raw_buffer_load_b128(s_w, (unsigned)(8 * kg * NN + 8 * ng) * 2u, (unsigned)((128 * q + r) * NN + 256 * jg) * 2u, 0);
            const int lbase = (kg >> 3) * (int)KTB + (ng >> 4) * (int)IMG;
#pragma unroll
            for (int jj = 0; jj < 8; ++jj) {
                u32x4 o;
#pragma unroll
                for (int d = 0; d < 4; ++d)
                    o[d] = (jj & 1) ? __builtin_amdgcn_perm(v[2 * d + 1][jj >> 1], v[2 * d][jj >> 1], 0x07060302u)
                                    : __builtin_amdgcn_perm(v[2 * d + 1][jj >> 1], v[2 * d][jj >> 1], 0x05040100u);
                const int R = 32 * ((ng & 15) >> 2) + 16 * (jj >> 2) + 4 * (ng & 3) + (jj & 3);
                *(LAS u32x4*)(lds + lbase + lds_byte(R, (kg & 7) * 8)) = o;
            }
            __syncthreads();
#pragma unroll
            for (int u = 0; u < 8; ++u) v[u] = *(const LAS u32x4*)(lds + u * 8192 + tid * 16);
            const __amdgpu_buffer_rsrc_t d_w = (jg & 7) < NPUSH ? mk_rsrc(P.ws[(jg >> 3) & 7], WS_USED) : d_ws;
#pragma unroll
            for (int u = 0; u < 8; ++u) ST128(v[u], d_w, dst + u * 8192u, 0, AUX_SYS);
        }
        WAIT_V(0);
        __syncthreads();
        const unsigned mask = cid < XPUSH ? (1u << rank) : (cid < NXP || ((cid - NXP) & 7) >= NSH) ? 0xffu : (1u << ((cid - NXP) >> 3));
        if (tid < 64) signal_ranks(P, mask, C_STG + rank * 512 + cid * 4 + q, call, lane);
        if (idx == g) STAMP(30);
    }
    if (tid == 0) __hip_atomic_fetch_add((unsigned*)P.ws[rank] + C_SDONE, 1u, __ATOMIC_RELAXED, __HIP_MEMORY_SCOPE_AGENT);
}

__device__ __forceinline__ unsigned cvt_pk_bf16(float lo, float hi) { unsigned r; asm volatile("v_cvt_pk_bf16_f32 %0, %1, %2" : "=v"(r) : "v"(lo), "v"(hi)); return r; }

__device__ __forceinline__ void gemm_worker(const Params& P, LAS unsigned char* lds, int g, unsigned call) {
    const int rank = P.rank, grp = g & 7;
    const int ntq = TAB.ntile[grp];
    if ((g >> 3) >= ntq) return;
    const int tid = threadIdx.x, wid = RFL(tid >> 6), lane = tid & 63, wr = wid >> 2, wc = wid & 3, fr = lane & 15, fq = lane >> 4;
    unsigned* ctl = (unsigned*)P.ws[rank];
    LAS unsigned* mbox = (LAS unsigned*)(lds + 8 * IMG);
    const char* wsb = P.ws[rank] + (size_t)((call & 1u) * PAR_STRIDE);
    const unsigned voff0 = (unsigned)tid * 16u, voff1 = voff0 + 8192u;
    const unsigned ldsw = (unsigned)wid * 1024u;
    const int aoff = lds_byte(wr * 64 + fr, fq * 8), boff = lds_byte(wc * 32 + fr, fq * 8);
    const int own8 = rank * 8;
    const unsigned ownoff = (unsigned)own8 * KTB;
#define SA(b, h) (((b) * 2 + (h)) * IMG)
#define SB(b, h) ((4 + (b) * 2 + (h)) * IMG)
#define STAGE(bufoff, gbase) do { \
        const char* _g = wsb + (size_t)(gbase); \
        __builtin_amdgcn_global_load_lds((const unsigned*)(_g + voff0), (LAS unsigned*)(lds + (bufoff) + ldsw), 16, 0, 0); \
        __builtin_amdgcn_global_load_lds((const unsigned*)(_g + voff1), (LAS unsigned*)(lds + (bufoff) + ldsw + 8192), 16, 0, 0); } while (0)
#define LDA(dst, b, h) do { _Pragma("unroll") for (int m = 0; m < 4; ++m) _Pragma("unroll") for (int k = 0; k < 2; ++k) dst[m][k] = *(const LAS bf16x8*)(lds + SA(b, h) + aoff + m * 2048 + k * 1024); } while (0)
#define LDB(dst, b, h) do { _Pragma("unroll") for (int n = 0; n < 2; ++n) _Pragma("unroll") for (int k = 0; k < 2; ++k) dst[n][k] = *(const LAS bf16x8*)(lds + SB(b, h) + boff + n * 2048 + k * 1024); } while (0)
#define MMA(ai, bj, At, Bt) do { __builtin_amdgcn_s_setprio(1); _Pragma("unroll") for (int m = 0; m < 4; ++m) _Pragma("unroll") for (int n = 0; n < 2; ++n) _Pragma("unroll") for (int k = 0; k < 2; ++k) \
        acc[ai][bj][m][n] = __builtin_amdgcn_mfma_f32_16x16x32_bf16(Bt[n][k], At[m][k], acc[ai][bj][m][n], 0, 0, 0); __builtin_amdgcn_s_setprio(0); } while (0)
#define ZERO_ACC() do { _Pragma("unroll") for (int a = 0; a < 2; ++a) _Pragma("unroll") for (int b = 0; b < 2; ++b) _Pragma("unroll") for (int m = 0; m < 4; ++m) _Pragma("unroll") for (int n = 0; n < 2; ++n) \
        acc[a][b][m][n] = (f32x4){0.f, 0.f, 0.f, 0.f}; } while (0)
    f32x4 acc[2][2][4][2];
    ZERO_ACC();
    bf16x8 At[4][2], B0[2][2], B1[2][2];
    int cur = TAB.tile[grp][g >> 3];
    int ti = cur & 15, tb = (cur >> 4) & 255, jg = wpanel(tb, rank);
    unsigned cA = OFF_X + (unsigned)ti * PANEL, cB = OFF_W + (unsigned)jg * PANEL;
    if (wid == 0) {
        if (lane == 0) mbox[4] = 0u;
        const u64 t0 = __builtin_amdgcn_s_memrealtime();
        const unsigned* w0 = ctl + C_STG + rank * 512 + ((lane & 4) ? NXP + jg : ti) * 4 + (lane & 3);
        for (;;) {
            const unsigned f = lane < 8 ? ld_agt(w0) : call;
            if (__all(f >= call)) break;
            if (expired(ctl, call, t0, lane)) break;
            __builtin_amdgcn_s_sleep(4);
        }
        __builtin_amdgcn_fence(__ATOMIC_ACQUIRE, "agent");
        WAIT_V(0); WAIT_L(0);
    }
    asm volatile("" ::: "memory"); BAR; asm volatile("" ::: "memory");
    STAMP(3);
    STAGE(SB(0, 0), cB + ownoff); STAGE(SA(0, 0), cA + ownoff); STAGE(SB(0, 1), cB + ownoff + IMG); STAGE(SA(0, 1), cA + ownoff + IMG);
    if (wr == 1) BAR;
    WAIT_V(4); BAR;
    STAGE(SB(1, 0), cB + ownoff + KTB); STAGE(SA(1, 0), cA + ownoff + KTB); STAGE(SB(1, 1), cB + ownoff + KTB + IMG);
    WAIT_V(6); BAR;
    unsigned a1 = cA + ownoff + KTB;
    int nt_done = 0;
    bool use_gq = false;
    for (;;) {
        int rho = 0, nxt = 0; unsigned bad = 0;
        unsigned nA = cA + ownoff, nB = cB + ownoff;
        const int mode = (cur >> 20) & 3, nkt = mode ? 32 : 64, th = mode ? 2 : 6, rlo = mode == 2 ? 28 : 0, rhi = mode == 1 ? 28 : 56;
        const unsigned own0 = ownoff + (mode == 2 ? 4u * KTB : 0u);
        for (int t = 0; t < nkt; t += 2) {
            if (t == th && wid == 0) {
                const __amdgpu_buffer_rsrc_t d_ws = mk_rsrc(ctl, 1u << 20);
                const int qw = use_gq ? C_GQ : C_TQ + 32 * grp;
                unsigned tk = 0;
                if (nt_done) tk = LANE0_ADD(d_ws, qw, lane);
                const bool wpushed = tb < NPUSH, xpushed = ti < XPUSH, pushed = wpushed || xpushed;
                const unsigned* w0 = lane >= 32 ? ctl + C_STG + ((lane >> 2) & 7) * 512 + (NXP + jg) * 4 + (lane & 3)
                                   : lane == 16 ? ctl + C_SDONE
                                   : (xpushed && lane < 8) ? ctl + C_XP + 32 * lane : ctl + C_ARR + ((lane & 8) ? NXP + jg : ti) * 8 + (lane & 7);
                const unsigned tgt = lane == 16 ? (unsigned)NWORK * call : (xpushed && lane < 8) ? 8u * call : call;
                const bool act = lane >= 32 ? wpushed : (lane == 16 || (lane < (wpushed ? 8 : 16) && (lane & 7) != rank));
                const u64 t0 = __builtin_amdgcn_s_memrealtime();
                if (nt_done < 4) STAMP(4 + 4 * nt_done);
                for (;;) {
                    const unsigned f = act ? ld_sys(w0) : tgt;
                    if (__all((int)(f - tgt) >= 0)) break;
                    if (expired(ctl, call, t0, lane)) break;
                    __builtin_amdgcn_s_sleep(8);
                }
                if (pushed) { __builtin_amdgcn_fence(__ATOMIC_ACQUIRE, ""); WAIT_V(0); }
                if (!nt_done) tk = LANE0_ADD(d_ws, qw, lane);
                int e = -1;
                if (!use_gq) {
                    const unsigned nx = (unsigned)WPG + (unsigned)RFL(tk) - (call - 1u) * (unsigned)ntq;
                    if (nx < (unsigned)ntq) e = TAB.tile[grp][nx];
                    else { use_gq = true; tk = LANE0_ADD(d_ws, C_GQ, lane); }
                }
                if (use_gq) {
                    const unsigned nx = (unsigned)RFL(tk) - (call - 1u) * (unsigned)(TAB.ngt + NACTIVE);
                    if (nx < (unsigned)TAB.ngt) e = TAB.gtile[nx];
                }
                const u64 now = __builtin_amdgcn_s_memrealtime();
                const unsigned tm = (now - t0 >= 100000ull) ? (unsigned)RFL(ld_agt(ctl + C_TMO)) : 0u;
                if (lane == 0) {
                    const unsigned ph = (unsigned)((now / (2 * KTICKS)) % 28ull);
                    mbox[0] = mode ? (unsigned)rlo + (ph % 14u) * 2u : ph * 2u;
                    mbox[1] = (unsigned)e;
                    mbox[2] = tm == call ? 1u : 0u;
                }
                if (nt_done < 4) STAMP(5 + 4 * nt_done);
            }
            LDB(B0, 0, 0); SCHED; LDA(At, 0, 0); STAGE(SA(1, 1), a1 + IMG);
            WAIT_L(8); BAR;
            u32x4 mb = {0u, 0u, 0u, 0u};
            if (t == th) mb = *(volatile LAS u32x4*)mbox;
            WAIT_L(0); MMA(0, 0, At, B0); BAR; SCHED;
            if (t == th) {
                rho = RFL((int)mb.x); nxt = RFL((int)mb.y); bad = (unsigned)RFL((int)mb.z);
                if (nxt >= 0) {
                    const unsigned no = ownoff + (((nxt >> 20) & 3) == 2 ? 4u * KTB : 0u);
                    nA = OFF_X + (unsigned)(nxt & 15) * PANEL + no; nB = OFF_W + (unsigned)wpanel((nxt >> 4) & 255, rank) * PANEL + no;
                }
            }
            unsigned a2, b2;
            if (t < th) { a2 = cA + own0 + (unsigned)(t + 2) * KTB; b2 = cB + own0 + (unsigned)(t + 2) * KTB; }
            else if (t < nkt - 2) { const unsigned ko = (unsigned)(rho + (rho >= own8 ? 8 : 0)) * KTB; a2 = cA + ko; b2 = cB + ko; rho = rho + 2 == rhi ? rlo : rho + 2; }
            else { a2 = nA; b2 = nB; }
            LDB(B1, 0, 1); STAGE(SB(0, 0), b2);
            BAR; WAIT_L(0); MMA(0, 1, At, B1); BAR;
            LDA(At, 0, 1); STAGE(SA(0, 0), a2);
            BAR; WAIT_L(0); MMA(1, 0, At, B0); BAR; SCHED;
            STAGE(SB(0, 1), b2 + IMG);
            WAIT_V(6); BAR; MMA(1, 1, At, B1); BAR;
            LDB(B0, 1, 0); SCHED; LDA(At, 1, 0); STAGE(SA(0, 1), a2 + IMG);
            WAIT_L(8); BAR; WAIT_L(0); MMA(0, 0, At, B0); BAR; SCHED;
            LDB(B1, 1, 1); STAGE(SB(1, 0), b2 + KTB);
            BAR; WAIT_L(0); MMA(0, 1, At, B1); BAR;
            LDA(At, 1, 1); STAGE(SA(1, 0), a2 + KTB);
            BAR; WAIT_L(0); MMA(1, 0, At, B0); BAR; SCHED;
            STAGE(SB(1, 1), b2 + KTB + IMG);
            WAIT_V(6); BAR; MMA(1, 1, At, B1); BAR;
            a1 = a2 + KTB;
        }
        if (nt_done < 4) STAMP(6 + 4 * nt_done);
        const __amdgpu_buffer_rsrc_t d_sl = mk_rsrc(P.ws[rank] + OFF_SLAB + (size_t)(cur >> 22) * SLAB, SLAB);
        unsigned* sflag = ctl + C_SLAB + (cur >> 22) * 8 + wid;
        unsigned tz = (unsigned)tid; asm volatile("" : "+v"(tz));
        const unsigned svo = tz * 16u;
        if (mode == 1) {
#pragma unroll
            for (int k = 0; k < 32; ++k) {
                u32x4 sv = __builtin_bit_cast(u32x4, acc[k >> 4][(k >> 1) & 1][(k >> 2) & 3][k & 1]);
                if (bad) sv = (u32x4){0x7fc00000u, 0x7fc00000u, 0x7fc00000u, 0x7fc00000u};
                ST128(sv, d_sl, svo, (unsigned)k * 8192u, AUX_AGT);
            }
            WAIT_V(0);
            if (lane == 0) st_agt(sflag, call);
        } else {
            const __amdgpu_buffer_rsrc_t d_out = mk_rsrc(P.out[rank], OUT_BYTES);
            const unsigned obase = (unsigned)(wr * 64 + (tz & 15u)) * OUT_PITCH + (unsigned)(wc * 32 + 8 * ((tz >> 4) & 3u)) * 2u;
            const unsigned sbase = (unsigned)(256 * ti) * OUT_PITCH + (unsigned)jg * 512u;
#define PUT(ai, m, bj, v0, v1) do { u32x4 wv; wv.x = cvt_pk_bf16(v0[0], v0[1]); wv.y = cvt_pk_bf16(v0[2], v0[3]); wv.z = cvt_pk_bf16(v1[0], v1[1]); wv.w = cvt_pk_bf16(v1[2], v1[3]); \
                if (bad) wv = (u32x4){0x7fc07fc0u, 0x7fc07fc0u, 0x7fc07fc0u, 0x7fc07fc0u}; \
                ST128(wv, d_out, obase + (bj) * 256u, sbase + (unsigned)((ai) * 128 + (m) * 16) * OUT_PITCH, AUX_SYS); } while (0)
            if (mode == 0) {
#pragma unroll
                for (int ai = 0; ai < 2; ++ai)
#pragma unroll
                    for (int m = 0; m < 4; ++m)
#pragma unroll
                        for (int bj = 0; bj < 2; ++bj) { const f32x4 v0 = acc[ai][bj][m][0], v1 = acc[ai][bj][m][1]; PUT(ai, m, bj, v0, v1); }
            } else {
                const u64 t0 = __builtin_amdgcn_s_memrealtime();
                for (;;) {
                    const unsigned f = (unsigned)RFL(ld_agt(sflag));
                    if (f >= call) break;
                    if (expired(ctl, call, t0, lane)) { bad = 1u; break; }
                    __builtin_amdgcn_s_sleep(4);
                }
#pragma unroll
                for (int g8 = 0; g8 < 4; ++g8) {
                    u32x4 pv[8];
#pragma unroll
                    for (int k = 0; k < 8; ++k) pv[k] = __builtin_amdgcn_raw_buffer_load_b128(d_sl, svo, (unsigned)(g8 * 8 + k) * 8192u, AUX_AGT);
#pragma unroll
                    for (int h = 0; h < 4; ++h) {
                        const int ai = g8 >> 1, m = (g8 & 1) * 2 + (h >> 1), bj = h & 1;
                        const f32x4 v0 = acc[ai][bj][m][0] + __builtin_bit_cast(f32x4, pv[2 * h]), v1 = acc[ai][bj][m][1] + __builtin_bit_cast(f32x4, pv[2 * h + 1]);
                        PUT(ai, m, bj, v0, v1);
                    }
                    SCHED;
                }
            }
#undef PUT
            if (tb < NSH) {
                WAIT_V(0);
                unsigned c = 0;
                if (lane == 0) c = __hip_atomic_fetch_add(mbox + 4, 1u, __ATOMIC_RELAXED, __HIP_MEMORY_SCOPE_WORKGROUP);
                if (((unsigned)RFL(c) & 7u) == 7u) signal_ranks(P, 0xffu & ~(1u << rank), C_YF + rank * 128 + ((cur >> 12) & 255), call, lane);
            }
        }
        if (nt_done < 4) STAMP(7 + 4 * nt_done);
        ++nt_done;
        if (nxt < 0) break;
        ZERO_ACC();
        cur = nxt; ti = nxt & 15; tb = (nxt >> 4) & 255; jg = wpanel(tb, rank);
        cA = OFF_X + (unsigned)ti * PANEL; cB = OFF_W + (unsigned)jg * PANEL;
    }
    WAIT_V(0);
    if (wr == 0) BAR;
#undef SA
#undef SB
#undef STAGE
#undef LDA
#undef LDB
#undef MMA
#undef ZERO_ACC
}

__global__ __launch_bounds__(512, 2) void fused_gemm_ar(Params P) {
    extern __shared__ __attribute__((aligned(16))) unsigned char shm[];
    LAS unsigned char* lds = (LAS unsigned char*)shm;
    const int b = blockIdx.x, rank = P.rank;
    unsigned* ctl = (unsigned*)P.ws[rank];
    const unsigned call = *(const CAS unsigned*)(u64)(ctl + C_FIN) / (unsigned)GRID + 1u;
    STAMP(0);
    if (b < NPULL) {
        puller(P, lds, b, call);
    } else {
        const int g = b - NPULL;
        stager(P, lds, g, call);
        STAMP(2);
        gemm_worker(P, lds, g, call);
    }
    __syncthreads();
    STAMP(1);
    STAMP(60 + (call & 1));
    if (threadIdx.x == 0) __hip_atomic_fetch_add(ctl + C_FIN, 1u, __ATOMIC_RELAXED, __HIP_MEMORY_SCOPE_AGENT);
}
}

extern "C" void kernel_launch(int rank, int nranks, ncclComm_t comm,
                              hipStream_t stream,
                              void* const* d_in, const int* in_sizes, int n_in,
                              void* d_out, int out_size,
                              const int* in_dtypes, int out_dtype,
                              void* const* d_out_all,
                              void* const* d_ws_all, size_t ws_bytes) {
    if (nranks != NR || (unsigned)rank >= (unsigned)NR || ws_bytes < WS_USED || n_in < 2 ||
        in_sizes[0] != MM * KS || in_sizes[1] != KS * NN || out_size != MM * NN) {
        fprintf(stderr, "kernel_launch: unexpected problem shape\n");
        return;
    }
    static unsigned attr_done = 0;
    int dev = 0; HIP_CHECK(hipGetDevice(&dev));
    if (!((attr_done >> (dev & 31)) & 1u)) {
        HIP_CHECK(hipFuncSetAttribute((const void*)fused_gemm_ar, hipFuncAttributeMaxDynamicSharedMemorySize, (int)LDS_BYTES));
        attr_done |= 1u << (dev & 31);
    }
    Params P;
    P.x = (const unsigned short*)d_in[0];
    P.w = (const unsigned short*)d_in[1];
    for (int p = 0; p < NR; ++p) { P.ws[p] = (char*)d_ws_all[p]; P.out[p] = (char*)d_out_all[p]; }
    P.rank = rank;
    hipLaunchKernelGGL(fused_gemm_ar, dim3(GRID), dim3(512), LDS_BYTES, stream, P);
    HIP_CHECK(hipGetLastError());
}
```

```cpp
#include <hip/hip_runtime.h>
#include <rccl/rccl.h>
#include <cstdio>
#include <cstdint>

#define HIP_CHECK(expr) do { hipError_t _e = (expr); if (_e != hipSuccess) { \
    fprintf(stderr, "HIP error %s at %s:%d\n", hipGetErrorString(_e), __FILE__, __LINE__); \
    return; } } while (0)

#define EREP 4
#define PUSH0 1
#define RTAIL 6
#define NPRE 48
#define PRIO1 40
#define PRIO2 112
#define DLY1 150
#define DLY2 300
#define SADDR 1
#define SLAB16 1
#define KTICKS 100

namespace {
#define LAS __attribute__((address_space(3)))
#define GAS __attribute__((address_space(1)))
#define CAS __attribute__((address_space(4)))
typedef short bf16x8 __attribute__((ext_vector_type(8)));
typedef float f32x4 __attribute__((ext_vector_type(4)));
typedef unsigned u32x4 __attribute__((ext_vector_type(4)));
typedef unsigned u32x2 __attribute__((ext_vector_type(2)));
typedef unsigned long long u64;

constexpr int NR = 8;
constexpr int MM = 4096, NN = 16384, KS = 512;
constexpr unsigned IMG = 16384;
constexpr unsigned KTB = 2 * IMG;
constexpr unsigned CHUNK = 8 * KTB;
constexpr unsigned PANEL = NR * CHUNK;
constexpr int NXP = MM / 256, NWP = NN / 256;
constexpr int WPR = NWP / NR;
constexpr int NSH = WPR - EREP;
constexpr int NW = NSH + NR * EREP;
constexpr unsigned OFF_X = 1u << 20;
constexpr unsigned OFF_W = OFF_X + NXP * PANEL;
constexpr unsigned PAR_STRIDE = (NXP + NWP) * PANEL;
constexpr unsigned OFF_SLAB = OFF_X + 2 * PAR_STRIDE;
constexpr unsigned SLAB = 256 * 256 * 4;
constexpr unsigned WS_USED = OFF_SLAB + 448 * SLAB;
constexpr unsigned OUT_BYTES = (unsigned)MM * NN * 2;
constexpr unsigned OUT_PITCH = NN * 2;
constexpr int NCHUNK = NXP + NWP;
constexpr int XPUSH = PUSH0 ? 1 : 0;
constexpr int NSEQ = NXP + NW - XPUSH;
constexpr int PRE_COST = 150, MAXSLAB = 448;
constexpr int NYT = NXP * NSH;
constexpr int NIN_IT = NSEQ * 64;
constexpr int NTOT_IT = NIN_IT + NYT * 64;
constexpr int GRID = 256, NPULL = 56, NWORK = GRID - NPULL, WPG = NWORK / 8;
constexpr int NJOB = NCHUNK * 4;
constexpr int C_FIN = 32, C_TMO = 64, C_SDONE = 96;
constexpr int C_XP = 128  ;
constexpr int C_Q = 512  , C_GQ = 736, C_TQ = 768  ;
constexpr int C_STG = 1024  ;
constexpr int C_YF = 8192  ;
constexpr int C_ARR = 12288  ;
constexpr int C_SLAB = 40960  ;
constexpr int C_ACNT = 16384  ;
constexpr unsigned LDS_BYTES = 8 * IMG + 256;
constexpr u64 DEADLINE = 40000000ull;
constexpr int AUX_SYS = 17;
constexpr int AUX_AGT = 16;
static_assert(RTAIL <= 16 && NSEQ <= 64 && NYT <= 128 && C_ACNT + 64 * 8 * 32 <= C_SLAB && (C_SLAB + 448 * 8) * 4 <= (512 << 10), "layout");

struct Params {
    const unsigned short* x;
    const unsigned short* w;
    char* ws[NR];
    char* out[NR];
    int rank;
};

constexpr int pos_x(int i) { return 2 * i; }
constexpr int pos_w(int b) { return b < NXP ? 2 * b + 1 : b + NXP; }
constexpr int tile_pos(int i, int b) { return pos_x(i) > pos_w(b) ? pos_x(i) : pos_w(b); }
constexpr int NSEQ_POS = (NW >= NXP) ? NXP + NW : 2 * NXP;
#define IT_I(e) ((e) & 15)
#define IT_B(e) (((e) >> 4) & 63)
#define IT_Y(e) (((e) >> 10) & 127)
#define IT_K(e) (((e) >> 17) & 7)
#define IT_S(e) ((e) >> 20)
constexpr int MAXPRE = 4;
struct Tables {
    int tile[8][96];
    int ntile[8];
    int pre[8][WPG][MAXPRE + 1];
    int gtile[2 * 8 * 16];
    int ngt, nslab;
    int ytile[128];
    int seq[64];
};
constexpr Tables make_tables() {
    Tables T{};
    int ny = 0, ns = 0;
    for (int pos = 0; pos < NSEQ_POS; ++pos) {
        for (int i = XPUSH; i < NXP; ++i) if (pos_x(i) == pos) T.seq[ns++] = i;
        for (int b = 0; b < NW; ++b) if (pos_w(b) == pos) T.seq[ns++] = -1 - b;
        for (int b = 0; b < NW; ++b)
            for (int i = 0; i < NXP; ++i)
                if (tile_pos(i, b) == pos) {
                    const int q = (i & 3) + 4 * (b & 1);
                    T.tile[q][T.ntile[q]++] = i | (b << 4) | (b < NSH ? ny << 10 : 0);
                    if (b < NSH) T.ytile[ny++] = i | (b << 4);
                }
    }
    int gt[8 * 16] = {}; int ng = 0;
    for (int k = RTAIL; k >= 1; --k)
        for (int q = 0; q < 8; ++q)
            if (T.ntile[q] - RTAIL >= WPG) gt[ng++] = T.tile[q][T.ntile[q] - k];
    for (int q = 0; q < 8; ++q)
        if (T.ntile[q] - RTAIL >= WPG) T.ntile[q] -= RTAIL;
    constexpr int LAG = 8;
    for (int k = 0; k < ng + LAG; ++k) {
        if (k < ng) T.gtile[T.ngt++] = gt[k] | (1 << 17) | (k << 20);
        if (k >= LAG) T.gtile[T.ngt++] = gt[k - LAG] | (2 << 17) | ((k - LAG) << 20);
    }
    T.nslab = ng;
    for (int q = 0; q < 8; ++q) {
        int np[WPG] = {}, slack[WPG] = {};
        for (int w = 0; w < WPG; ++w) {
            for (int k = 0; k <= MAXPRE; ++k) T.pre[q][w][k] = -1;
            slack[w] = w < T.ntile[q] ? 60 + 44 * (tile_pos(IT_I(T.tile[q][w]), IT_B(T.tile[q][w])) + 1 - XPUSH) - 240 : 0;
        }
        for (int k = WPG; k < T.ntile[q] && k < WPG + NPRE; ++k) {
            int best = 0;
            for (int w = 1; w < WPG; ++w) if (slack[w] > slack[best]) best = w;
            if (slack[best] < PRE_COST || np[best] >= MAXPRE) break;
            T.pre[q][best][np[best]++] = T.tile[q][k] | (3 << 17) | (T.nslab << 20);
            T.tile[q][k] |= (4 << 17) | (T.nslab << 20);
            ++T.nslab; slack[best] -= PRE_COST;
            if (np[best] >= MAXPRE) slack[best] = 0;
        }
    }
    return T;
}
__constant__ const Tables TAB = make_tables();
constexpr int n_active() { const Tables T = make_tables(); int n = 0; for (int q = 0; q < 8; ++q) n += T.ntile[q] < WPG ? T.ntile[q] : WPG; return n; }
constexpr int NACTIVE = n_active();
static_assert(make_tables().nslab <= MAXSLAB, "slabs");

__device__ __forceinline__ int lds_byte(int r, int c) { const int st = (r >> 4) * 2 + (c >> 5), rr = r & 15, cc = c & 31, ob = rr * 64 + cc * 2; return st * 1024 + (ob ^ (((ob >> 9) & 1) << 5)); }
__device__ __forceinline__ void stage_rc(int b, int& R, int& C) { const int st = b / 1024, sb = b % 1024, swz = sb ^ (((sb >> 9) & 1) << 5); R = (st >> 1) * 16 + swz / 64; C = (st & 1) * 32 + (swz % 64) / 2; }
__device__ __forceinline__ int wpanel(int b, int r) { const int m = b - NSH; return b < NSH ? r * WPR + b : (m & 7) * WPR + NSH + (m >> 3); }
__device__ __forceinline__ int seq_chunk(int s, int r) { const int v = TAB.seq[s]; return v >= 0 ? v : NXP + wpanel(-1 - v, r); }
__device__ __forceinline__ unsigned chunk_off(int cid, int src, unsigned call) { return OFF_X + (call & 1u) * PAR_STRIDE + (unsigned)cid * PANEL + (unsigned)src * CHUNK; }

__device__ __forceinline__ __amdgpu_buffer_rsrc_t mk_rsrc(const void* p, unsigned bytes) {
    const u64 a = (u64)p;
    const unsigned lo = __builtin_amdgcn_readfirstlane((unsigned)a), hi = __builtin_amdgcn_readfirstlane((unsigned)(a >> 32));
    return __builtin_amdgcn_make_buffer_rsrc((void*)(((u64)hi << 32) | lo), 0, bytes, 0x00020000);
}
__device__ __forceinline__ unsigned ld_sys(const unsigned* p) { return __hip_atomic_load(p, __ATOMIC_RELAXED, __HIP_MEMORY_SCOPE_SYSTEM); }
__device__ __forceinline__ unsigned ld_agt(const unsigned* p) { return __hip_atomic_load(p, __ATOMIC_RELAXED, __HIP_MEMORY_SCOPE_AGENT); }
__device__ __forceinline__ void st_sys(unsigned* p, unsigned v) { __hip_atomic_store(p, v, __ATOMIC_RELAXED, __HIP_MEMORY_SCOPE_SYSTEM); }
__device__ __forceinline__ void st_agt(unsigned* p, unsigned v) { __hip_atomic_store(p, v, __ATOMIC_RELAXED, __HIP_MEMORY_SCOPE_AGENT); }
#define RFL(x) __builtin_amdgcn_readfirstlane(x)
#define ST128(v, rsrc, voff, soff, aux) __builtin_amdgcn_raw_buffer_store_b128(v, rsrc, (voff) + (soff), 0, aux)
#define STAMP(slot) do { } while (0)
#define WAIT_V(n) asm volatile("s_waitcnt vmcnt(" #n ")" ::: "memory")
#define WAIT_L(n) asm volatile("s_waitcnt lgkmcnt(" #n ")" ::: "memory")
#define BAR __builtin_amdgcn_s_barrier()
#define SCHED __builtin_amdgcn_sched_barrier(0)
#define LANE0_ADD(rsrc, word, lane) (unsigned)__builtin_amdgcn_raw_ptr_buffer_atomic_add_i32(1, rsrc, (lane) == 0 ? (int)((word) * 4) : (int)0xfffffff0u, 0, 0)

__device__ __forceinline__ void signal_ranks(const Params& P, unsigned mask, int word, unsigned val, int lane) {
    const CAS Params* kp = (const CAS Params*)__builtin_amdgcn_kernarg_segment_ptr();
    asm volatile("" : "+s"(kp));
    u64 a = 0;
#pragma unroll
    for (int p = 0; p < NR; ++p) { u64 t = (u64)kp->ws[p]; asm volatile("" : "+s"(t)); a = (lane == p) ? t : a; }
    if (lane < NR && ((mask >> lane) & 1u)) __hip_atomic_store((GAS unsigned*)a + word, val, __ATOMIC_RELAXED, __HIP_MEMORY_SCOPE_SYSTEM);
}
__device__ __forceinline__ bool expired(unsigned* ctl, unsigned call, u64 t0, int lane) {
    const u64 dt = __builtin_amdgcn_s_memrealtime() - t0;
    if (dt < 100000ull) return false;
    const bool dead = dt > DEADLINE || (unsigned)RFL(ld_agt(ctl + C_TMO)) == call;
    if (dead && lane == 0) st_agt(ctl + C_TMO, call);
    return RFL((int)dead) != 0;
}

__device__ __forceinline__ void puller(const Params& P, LAS unsigned char* lds, int b, unsigned call) {
    const int rank = P.rank, lane = threadIdx.x & 63, link = b >> 3;
    const unsigned p = (unsigned)(rank + 1 + link) & 7u;
    unsigned* ctl = (unsigned*)P.ws[rank];
    const __amdgpu_buffer_rsrc_t s_ws = mk_rsrc(P.ws[p], WS_USED), d_ws = mk_rsrc(P.ws[rank], WS_USED);
    const __amdgpu_buffer_rsrc_t s_out = mk_rsrc(P.out[p], OUT_BYTES), d_out = mk_rsrc(P.out[rank], OUT_BYTES);
    const unsigned qbase = (call - 1u) * (unsigned)(NTOT_IT + 64);
    const int qword = C_Q + 32 * link;
    bool dead = false;
    bool xp_pend = false;
    LAS unsigned* xcnt = (LAS unsigned*)lds;
    if (XPUSH) {
        const int k = (b & 7) * 8 + RFL((int)(threadIdx.x >> 6));
        const __amdgpu_buffer_rsrc_t s_x = mk_rsrc(P.x, MM * KS * 2);
        u32x4 v[4];
#pragma unroll
        for (int q = 0; q < 4; ++q) {
            int R, C; stage_rc((k & 3) * 4096 + q * 1024 + lane * 16, R, C);
            v[q] = __builtin_amdgcn_raw_buffer_load_b128(s_x, (unsigned)(R * KS + C) * 2u, (unsigned)(128 * ((k >> 2) & 1) * KS + 64 * (k >> 3)) * 2u, 0);
        }
        if (threadIdx.x == 0) *xcnt = 0u;
        __syncthreads();
#pragma unroll
        for (int q = 0; q < 4; ++q) ST128(v[q], s_ws, (unsigned)lane * 16u, chunk_off(0, rank, call) + (unsigned)k * 4096u + q * 1024u, AUX_SYS);
        xp_pend = true;
    }
#define XP_DONE() do { if (XPUSH && xp_pend) { xp_pend = false; \
        if (lane == 0 && __hip_atomic_fetch_add(xcnt, 1u, __ATOMIC_RELAXED, __HIP_MEMORY_SCOPE_WORKGROUP) == 7u) \
            __hip_atomic_fetch_add((unsigned*)P.ws[p] + C_XP + 32 * rank, 1u, __ATOMIC_RELAXED, __HIP_MEMORY_SCOPE_SYSTEM); } } while (0)
    unsigned tk = LANE0_ADD(d_ws, qword, lane);
    const int mycid = seq_chunk(lane < NSEQ ? lane : 0, rank);
    const unsigned ytab = (unsigned)TAB.ytile[lane] | ((unsigned)TAB.ytile[64 + lane] << 8);
    unsigned nn = (unsigned)RFL(tk) - qbase;
    int pend = -1;
    {
        const unsigned foff = (unsigned)(C_STG + p * 512 + mycid * 4) * 4u;
        u64 rq0 = 0, rq1 = 0, rq2 = 0, rq3 = 0;
        while (nn < (unsigned)NIN_IT) {
            const unsigned n = nn;
            const int s = (int)(n >> 6), qq = (int)(n >> 4) & 3;
            bool rdy = (((qq == 0 ? rq0 : qq == 1 ? rq1 : qq == 2 ? rq2 : rq3) >> s) & 1) != 0;
            if (!rdy && !dead) {
                const u64 t0 = __builtin_amdgcn_s_memrealtime();
                for (;;) {
                    const u32x4 f = __builtin_amdgcn_raw_buffer_load_b128(d_ws, foff, 0, AUX_SYS);
                    rq0 = __ballot(f.x >= call); rq1 = __ballot(f.y >= call); rq2 = __ballot(f.z >= call); rq3 = __ballot(f.w >= call);
                    if (((qq == 0 ? rq0 : qq == 1 ? rq1 : qq == 2 ? rq2 : rq3) >> s) & 1) break;
                    if (xp_pend) { WAIT_V(0); XP_DONE(); }
                    if (pend >= 0) {
                        WAIT_V(0);
                        const unsigned c = (unsigned)RFL(LANE0_ADD(d_ws, C_ACNT + (pend * 8 + (int)p) * 32, lane));
                        if ((c & 63u) == 63u && lane == 0) st_agt(ctl + C_ARR + __builtin_amdgcn_readlane(mycid, pend) * 8 + p, call);
                        pend = -1;
                    }
                    if (expired(ctl, call, t0, lane)) { dead = true; break; }
                    __builtin_amdgcn_s_sleep(1);
                }
            }
            const int cid = __builtin_amdgcn_readlane(mycid, s);
            const unsigned soff = chunk_off(cid, (int)p, call) + (n & 63u) * 4096u, voff = (unsigned)lane * 16u;
            tk = LANE0_ADD(d_ws, qword, lane);
            u32x4 v = {0u, 0u, 0u, 0u};
            if (!dead) v = __builtin_amdgcn_raw_buffer_load_b128(s_ws, voff, soff, AUX_SYS);
            WAIT_V(0);
            nn = (unsigned)RFL(tk) - qbase;
            ST128(v, d_ws, voff, soff, AUX_AGT);
            XP_DONE();
            unsigned c = 0;
            if (pend >= 0) c = LANE0_ADD(d_ws, C_ACNT + (pend * 8 + (int)p) * 32, lane);
            if (!dead) v = __builtin_amdgcn_raw_buffer_load_b128(s_ws, voff, soff + 1024u, AUX_SYS);
            WAIT_V(0);
            ST128(v, d_ws, voff, soff + 1024u, AUX_AGT);
            if (pend >= 0 && ((unsigned)RFL(c) & 63u) == 63u && lane == 0) st_agt(ctl + C_ARR + __builtin_amdgcn_readlane(mycid, pend) * 8 + p, call);
            if (!dead) v = __builtin_amdgcn_raw_buffer_load_b128(s_ws, voff, soff + 2048u, AUX_SYS);
            ST128(v, d_ws, voff, soff + 2048u, AUX_AGT);
            if (!dead) v = __builtin_amdgcn_raw_buffer_load_b128(s_ws, voff, soff + 3072u, AUX_SYS);
            ST128(v, d_ws, voff, soff + 3072u, AUX_AGT);
            pend = s;
        }
        if (xp_pend) { WAIT_V(0); XP_DONE(); }
        if (pend >= 0) {
            WAIT_V(0);
            const unsigned c = (unsigned)RFL(LANE0_ADD(d_ws, C_ACNT + (pend * 8 + (int)p) * 32, lane));
            if ((c & 63u) == 63u && lane == 0) st_agt(ctl + C_ARR + __builtin_amdgcn_readlane(mycid, pend) * 8 + p, call);
        }
    }
    {
        const unsigned* fl = ctl + C_YF + p * 128 + lane;
        const unsigned voff = (unsigned)(lane >> 5) * OUT_PITCH + p * (WPR * 512u) + (unsigned)(lane & 31) * 16u;
        u64 r0 = 0, r1 = 0;
        while (nn < (unsigned)NTOT_IT) {
            const unsigned n = nn - (unsigned)NIN_IT;
            const int o = (int)(n >> 6);
            if (!dead && !(((o < 64 ? r0 : r1) >> (o & 63)) & 1)) {
                const u64 t0 = __builtin_amdgcn_s_memrealtime();
                for (;;) {
                    const unsigned f0 = ld_sys(fl), f1 = ld_sys(fl + 64);
                    r0 = __ballot(f0 >= call); r1 = __ballot(f1 >= call);
                    if (((o < 64 ? r0 : r1) >> (o & 63)) & 1) break;
                    if (expired(ctl, call, t0, lane)) { dead = true; break; }
                    __builtin_amdgcn_s_sleep(1);
                }
            }
            const unsigned e = ((unsigned)__builtin_amdgcn_readlane((int)ytab, o & 63) >> ((o >> 6) * 8)) & 255u;
            const unsigned soff = ((e & 15u) * 256u + (n & 63u) * 4u) * OUT_PITCH + (e >> 4) * 512u;
            if (!dead) {
                const u32x4 v0 = __builtin_amdgcn_raw_buffer_load_b128(s_out, voff, soff, AUX_SYS);
                tk = LANE0_ADD(d_ws, qword, lane);
                const int o2 = o + 3 < NYT ? o + 3 : NYT - 1;
                const bool refresh = !(((o2 < 64 ? r0 : r1) >> (o2 & 63)) & 1);
                unsigned f0 = 0, f1 = 0;
                if (refresh) { f0 = ld_sys(fl); f1 = ld_sys(fl + 64); }
                ST128(v0, d_out, voff, soff, AUX_AGT);
                const u32x4 v1 = __builtin_amdgcn_raw_buffer_load_b128(s_out, voff, soff + 2 * OUT_PITCH, AUX_SYS);
                nn = (unsigned)RFL(tk) - qbase;
                if (refresh) { r0 = __ballot(f0 >= call); r1 = __ballot(f1 >= call); }
                SCHED;
                ST128(v1, d_out, voff, soff + 2 * OUT_PITCH, AUX_AGT);
            } else {
                tk = LANE0_ADD(d_ws, qword, lane);
                nn = (unsigned)RFL(tk) - qbase;
                const u32x4 nanv = {0x7fc07fc0u, 0x7fc07fc0u, 0x7fc07fc0u, 0x7fc07fc0u};
                ST128(nanv, d_out, voff, soff, AUX_AGT);
                ST128(nanv, d_out, voff, soff + 2 * OUT_PITCH, AUX_AGT);
            }
        }
    }
}

__device__ __forceinline__ void job_of(int idx, int& cid, int& q) {
    q = idx & 3;
    int u = idx >> 2;
    if (u < 9 * NSH) { const int g = u / 9, r = u % 9; cid = r < 8 ? NXP + r * WPR + g : (g == 0 ? XPUSH : g == 1 ? 1 - XPUSH : g); return; }
    u -= 9 * NSH;
    constexpr int NXR = NXP - NSH, NPAIR = NXR < 8 * EREP ? NXR : 8 * EREP;
    if (u < 2 * NPAIR) { cid = (u & 1) ? NXP + wpanel(NSH + (u >> 1), 0) : NSH + (u >> 1); return; }
    u -= 2 * NPAIR;
    cid = NXR > NPAIR ? NSH + NPAIR + u : NXP + wpanel(NSH + NPAIR + u, 0);
}
__device__ __forceinline__ void stager(const Params& P, LAS unsigned char* lds, int g, unsigned call) {
    const int rank = P.rank, tid = threadIdx.x, lane = tid & 63;
    const __amdgpu_buffer_rsrc_t d_ws = mk_rsrc(P.ws[rank], WS_USED);
    const __amdgpu_buffer_rsrc_t s_x = mk_rsrc(P.x, MM * KS * 2), s_w = mk_rsrc(P.w, KS * NN * 2);
    if (g >= PRIO1) {
        const u64 t0 = __builtin_amdgcn_s_memrealtime(), dly = g >= PRIO2 ? DLY2 : DLY1;
        while (__builtin_amdgcn_s_memrealtime() - t0 < dly) __builtin_amdgcn_s_sleep(4);
    }
    for (int idx = g; idx < NJOB; idx += NWORK) {
        int cid, q; job_of(idx, cid, q);
        const unsigned dst = chunk_off(cid, rank, call) + (unsigned)q * (2 * KTB) + (unsigned)tid * 16u;
        if (cid < NXP) {
            u32x4 v[8];
#pragma unroll
            for (int u = 0; u < 8; ++u) {
                int R, C; stage_rc((u & 1) * 8192 + tid * 16, R, C);
                v[u] = __builtin_amdgcn_raw_buffer_load_b128(s_x, (unsigned)(R * KS + C) * 2u, (unsigned)((256 * cid + 128 * ((u >> 1) & 1)) * KS + 128 * q + 64 * (u >> 2)) * 2u, 0);
            }
#pragma unroll
            for (int u = 0; u < 8; ++u) ST128(v[u], d_ws, dst + u * 8192u, 0, AUX_SYS);
        } else {
            const int jg = cid - NXP, ng = tid & 31, kg = tid >> 5;
            u32x4 v[8];
#pragma unroll
            for (int r = 0; r < 8; ++r) v[r] = __builtin_amdgcn_raw_buffer_load_b128(s_w, (unsigned)(8 * kg * NN + 8 * ng) * 2u, (unsigned)((128 * q + r) * NN + 256 * jg) * 2u, 0);
            const int lbase = (kg >> 3) * (int)KTB + (ng >> 4) * (int)IMG;
#pragma unroll
            for (int jj = 0; jj < 8; ++jj) {
                u32x4 o;
#pragma unroll
                for (int d = 0; d < 4; ++d)
                    o[d] = (jj & 1) ? __builtin_amdgcn_perm(v[2 * d + 1][jj >> 1], v[2 * d][jj >> 1], 0x07060302u)
                                    : __builtin_amdgcn_perm(v[2 * d + 1][jj >> 1], v[2 * d][jj >> 1], 0x05040100u);
                const int R = 32 * ((ng & 15) >> 2) + 16 * (jj >> 2) + 4 * (ng & 3) + (jj & 3);
                *(LAS u32x4*)(lds + lbase + lds_byte(R, (kg & 7) * 8)) = o;
            }
            __syncthreads();
#pragma unroll
            for (int u = 0; u < 8; ++u) v[u] = *(const LAS u32x4*)(lds + u * 8192 + tid * 16);
#pragma unroll
            for (int u = 0; u < 8; ++u) ST128(v[u], d_ws, dst + u * 8192u, 0, AUX_SYS);
        }
        WAIT_V(0);
        __syncthreads();
        const unsigned mask = cid < XPUSH ? (1u << rank) : (cid < NXP || ((cid - NXP) & 7) >= NSH) ? 0xffu : (1u << ((cid - NXP) >> 3));
        if (tid < 64) signal_ranks(P, mask, C_STG + rank * 512 + cid * 4 + q, call, lane);
        if (idx == g) STAMP(30);
    }
    if (tid == 0) __hip_atomic_fetch_add((unsigned*)P.ws[rank] + C_SDONE, 1u, __ATOMIC_RELAXED, __HIP_MEMORY_SCOPE_AGENT);
}

__device__ __forceinline__ unsigned cvt_pk_bf16(float lo, float hi) { unsigned r; asm volatile("v_cvt_pk_bf16_f32 %0, %1, %2" : "=v"(r) : "v"(lo), "v"(hi)); return r; }

__device__ __forceinline__ unsigned wait_inputs(unsigned* ctl, unsigned call, int rank, int lane, bool arr, int i, int jg, bool sd, int slab) {
    const bool xp = i < XPUSH;
    const unsigned* w0 = lane >= 24 ? ctl + C_SLAB + (slab < 0 ? 0 : slab) * 8 + (lane & 7)
                       : lane == 16 ? ctl + C_SDONE
                       : (xp && lane < 8) ? ctl + C_XP + 32 * lane : ctl + C_ARR + ((lane & 8) ? NXP + jg : i) * 8 + (lane & 7);
    const unsigned tgt = lane == 16 ? (unsigned)NWORK * call : (xp && lane < 8) ? 8u * call : call;
    const bool act = lane < 16 ? (arr && (lane & 7) != rank) : lane == 16 ? sd : (lane >= 24 && lane < 32 && slab >= 0);
    const u64 t0 = __builtin_amdgcn_s_memrealtime();
    for (;;) {
        const unsigned f = act ? ld_sys(w0) : tgt;
        if (__all((int)(f - tgt) >= 0)) break;
        if (expired(ctl, call, t0, lane)) break;
        __builtin_amdgcn_s_sleep(8);
    }
    if (arr && xp) { __builtin_amdgcn_fence(__ATOMIC_ACQUIRE, ""); WAIT_V(0); }
    return (__builtin_amdgcn_s_memrealtime() - t0 >= 100000ull && (unsigned)RFL(ld_agt(ctl + C_TMO)) == call) ? 1u : 0u;
}

__device__ __forceinline__ void gemm_worker(const Params& P, LAS unsigned char* lds, int g, unsigned call) {
    const int rank = P.rank, grp = g & 7;
    const int ntq = TAB.ntile[grp];
    if ((g >> 3) >= ntq) return;
    const int tid = threadIdx.x, wid = RFL(tid >> 6), lane = tid & 63, wr = wid >> 2, wc = wid & 3, fr = lane & 15, fq = lane >> 4;
    unsigned* ctl = (unsigned*)P.ws[rank];
    LAS unsigned* mbox = (LAS unsigned*)(lds + 8 * IMG);
    const char* wsb = P.ws[rank] + (size_t)((call & 1u) * PAR_STRIDE);
    const unsigned voff0 = (unsigned)tid * 16u, voff1 = voff0 + 8192u;
    const unsigned ldsw = (unsigned)wid * 1024u;
    const int aoff = lds_byte(wr * 64 + fr, fq * 8), boff = lds_byte(wc * 32 + fr, fq * 8);
    const int own8 = rank * 8;
    const unsigned ownoff = (unsigned)own8 * KTB;
#define SA(b, h) (((b) * 2 + (h)) * IMG)
#define SB(b, h) ((4 + (b) * 2 + (h)) * IMG)
    const __amdgpu_buffer_rsrc_t s_pan = mk_rsrc(wsb, OFF_X + PAR_STRIDE);
#define STAGE(bufoff, gbase) do { \
        __builtin_amdgcn_raw_ptr_buffer_load_lds(s_pan, (LAS void*)(lds + (bufoff) + ldsw), 16, voff0, (gbase), 0, 0); \
        __builtin_amdgcn_raw_ptr_buffer_load_lds(s_pan, (LAS void*)(lds + (bufoff) + ldsw + 8192), 16, voff1, (gbase), 0, 0); } while (0)
#define LDA(dst, b, h) do { _Pragma("unroll") for (int m = 0; m < 4; ++m) _Pragma("unroll") for (int k = 0; k < 2; ++k) dst[m][k] = *(const LAS bf16x8*)(lds + SA(b, h) + aoff + m * 2048 + k * 1024); } while (0)
#define LDB(dst, b, h) do { _Pragma("unroll") for (int n = 0; n < 2; ++n) _Pragma("unroll") for (int k = 0; k < 2; ++k) dst[n][k] = *(const LAS bf16x8*)(lds + SB(b, h) + boff + n * 2048 + k * 1024); } while (0)
#define MMA(ai, bj, At, Bt) do { __builtin_amdgcn_s_setprio(1); _Pragma("unroll") for (int m = 0; m < 4; ++m) _Pragma("unroll") for (int n = 0; n < 2; ++n) _Pragma("unroll") for (int k = 0; k < 2; ++k) \
        acc[ai][bj][m][n] = __builtin_amdgcn_mfma_f32_16x16x32_bf16(Bt[n][k], At[m][k], acc[ai][bj][m][n], 0, 0, 0); __builtin_amdgcn_s_setprio(0); } while (0)
#define ZERO_ACC() do { _Pragma("unroll") for (int a = 0; a < 2; ++a) _Pragma("unroll") for (int b = 0; b < 2; ++b) _Pragma("unroll") for (int m = 0; m < 4; ++m) _Pragma("unroll") for (int n = 0; n < 2; ++n) \
        acc[a][b][m][n] = (f32x4){0.f, 0.f, 0.f, 0.f}; } while (0)
    f32x4 acc[2][2][4][2];
    ZERO_ACC();
    bf16x8 At[4][2], B0[2][2], B1[2][2];
    const int slot = g >> 3;
    int sidx = 0;
    int cur = TAB.pre[grp][slot][0];
    if (cur < 0) cur = TAB.tile[grp][slot];
    bool dyn = IT_K(cur) != 3;
    unsigned cA = OFF_X + (unsigned)IT_I(cur) * PANEL, cB = OFF_W + (unsigned)wpanel(IT_B(cur), rank) * PANEL;
    if (wid == 0) {
        if (lane == 0) mbox[4] = 0u;
        const u64 t0 = __builtin_amdgcn_s_memrealtime();
        const unsigned* w0 = ctl + C_STG + rank * 512 + ((lane & 4) ? NXP + wpanel(IT_B(cur), rank) : IT_I(cur)) * 4 + (lane & 3);
        for (;;) {
            const unsigned f = lane < 8 ? ld_agt(w0) : call;
            if (__all(f >= call)) break;
            if (expired(ctl, call, t0, lane)) break;
            __builtin_amdgcn_s_sleep(4);
        }
        __builtin_amdgcn_fence(__ATOMIC_ACQUIRE, "agent");
        WAIT_V(0); WAIT_L(0);
    }
    asm volatile("" ::: "memory"); BAR; asm volatile("" ::: "memory");
    STAMP(3);
    STAGE(SB(0, 0), cB + ownoff); STAGE(SA(0, 0), cA + ownoff); STAGE(SB(0, 1), cB + ownoff + IMG); STAGE(SA(0, 1), cA + ownoff + IMG);
    if (wr == 1) BAR;
    WAIT_V(4); BAR;
    STAGE(SB(1, 0), cB + ownoff + KTB); STAGE(SA(1, 0), cA + ownoff + KTB); STAGE(SB(1, 1), cB + ownoff + KTB + IMG);
    WAIT_V(6); BAR;
    unsigned a1 = cA + ownoff + KTB;
    int nt_done = 0, rho = 0;
    unsigned bad = 0;
    bool use_gq = false, drew = false;
    for (;;) {
        const int kind = IT_K(cur);
        const bool half = kind == 1 || kind == 2;
        const int nown = kind == 4 ? 0 : half ? 4 : 8, nkt = nown + (kind == 3 ? 0 : half ? 28 : 56);
        const int th = (kind == 0 || kind == 3) ? 6 : 2, rlo = kind == 2 ? 28 : 0, rhi = kind == 1 ? 28 : 56;
        const unsigned own0 = ownoff + (kind == 2 ? 4u * KTB : 0u);
        int nxt = -1; unsigned nbad = 0;
        unsigned nA = cA, nB = cB;
        for (int t = 0; t < nkt; t += 2) {
            const bool hookA = t == th, hookB = t == nkt - 2 && nxt >= 0 && IT_K(nxt) == 4;
            if (hookA && wid == 0) {
                const __amdgpu_buffer_rsrc_t d_ws = mk_rsrc(ctl, 1u << 20);
                const int qw = use_gq ? C_GQ : C_TQ + 32 * grp;
                unsigned tk = 0;
                if (dyn && drew) tk = LANE0_ADD(d_ws, qw, lane);
                if (nt_done < 6) STAMP(4 + 4 * nt_done);
                unsigned tm = 0;
                if (kind != 4) tm = wait_inputs(ctl, call, rank, lane, kind != 3, IT_I(cur), wpanel(IT_B(cur), rank), !drew, -1);
                int e = -1;
                if (!dyn) {
                    e = TAB.pre[grp][slot][++sidx];
                    if (e < 0) e = TAB.tile[grp][slot];
                } else {
                    if (!drew) { tk = LANE0_ADD(d_ws, qw, lane); drew = true; }
                    if (!use_gq) {
                        const unsigned nx = (unsigned)WPG + (unsigned)RFL(tk) - (call - 1u) * (unsigned)ntq;
                        if (nx < (unsigned)ntq) e = TAB.tile[grp][nx];
                        else { use_gq = true; tk = LANE0_ADD(d_ws, C_GQ, lane); }
                    }
                    if (use_gq) {
                        const unsigned nx = (unsigned)RFL(tk) - (call - 1u) * (unsigned)(TAB.ngt + NACTIVE);
                        if (nx < (unsigned)TAB.ngt) e = TAB.gtile[nx];
                    }
                }
                asm volatile("" :: "v"(tk));
                if (lane == 0) {
                    const unsigned ph = (unsigned)((__builtin_amdgcn_s_memrealtime() / (2 * KTICKS)) % 28ull);
                    mbox[0] = half ? (unsigned)rlo + (ph % 14u) * 2u : ph * 2u;
                    mbox[1] = (unsigned)e;
                    mbox[2] = tm;
                }
                if (nt_done < 6) STAMP(5 + 4 * nt_done);
            }
            if (hookB && wid == 0) {
                const unsigned tm = wait_inputs(ctl, call, rank, lane, true, IT_I(nxt), wpanel(IT_B(nxt), rank), false, IT_S(nxt));
                if (lane == 0) {
                    mbox[0] = (unsigned)((__builtin_amdgcn_s_memrealtime() / (2 * KTICKS)) % 28ull) * 2u;
                    mbox[2] = tm;
                }
            }
            LDB(B0, 0, 0); SCHED; LDA(At, 0, 0); STAGE(SA(1, 1), a1 + IMG);
            WAIT_L(8); BAR;
            u32x4 mb = {0u, 0u, 0u, 0u};
            if (hookA || hookB) mb = *(volatile LAS u32x4*)mbox;
            WAIT_L(0); MMA(0, 0, At, B0); BAR; SCHED;
            if (hookA) {
                if (kind != 4) { rho = RFL((int)mb.x); bad = (unsigned)RFL((int)mb.z); }
                nxt = RFL((int)mb.y);
                if (nxt >= 0) { nA = OFF_X + (unsigned)IT_I(nxt) * PANEL; nB = OFF_W + (unsigned)wpanel(IT_B(nxt), rank) * PANEL; }
            }
            unsigned a2, b2;
            if (t + 2 < nown) { a2 = cA + own0 + (unsigned)(t + 2) * KTB; b2 = cB + own0 + (unsigned)(t + 2) * KTB; }
            else if (t + 2 < nkt) { const unsigned ko = (unsigned)(rho + (rho >= own8 ? 8 : 0)) * KTB; a2 = cA + ko; b2 = cB + ko; rho = rho + 2 == rhi ? rlo : rho + 2; }
            else if (hookB) {
                rho = RFL((int)mb.x); nbad = (unsigned)RFL((int)mb.z);
                const unsigned ko = (unsigned)(rho + (rho >= own8 ? 8 : 0)) * KTB; a2 = nA + ko; b2 = nB + ko; rho = rho == 54 ? 0 : rho + 2;
            } else { const unsigned no = ownoff + ((nxt >= 0 && IT_K(nxt) == 2) ? 4u * KTB : 0u); a2 = nA + no; b2 = nB + no; }
            LDB(B1, 0, 1); STAGE(SB(0, 0), b2);
            BAR; WAIT_L(0); MMA(0, 1, At, B1); BAR;
            LDA(At, 0, 1); STAGE(SA(0, 0), a2);
            BAR; WAIT_L(0); MMA(1, 0, At, B0); BAR; SCHED;
            STAGE(SB(0, 1), b2 + IMG);
            WAIT_V(6); BAR; MMA(1, 1, At, B1); BAR;
            LDB(B0, 1, 0); SCHED; LDA(At, 1, 0); STAGE(SA(0, 1), a2 + IMG);
            WAIT_L(8); BAR; WAIT_L(0); MMA(0, 0, At, B0); BAR; SCHED;
            LDB(B1, 1, 1); STAGE(SB(1, 0), b2 + KTB);
            BAR; WAIT_L(0); MMA(0, 1, At, B1); BAR;
            LDA(At, 1, 1); STAGE(SA(1, 0), a2 + KTB);
            BAR; WAIT_L(0); MMA(1, 0, At, B0); BAR; SCHED;
            STAGE(SB(1, 1), b2 + KTB + IMG);
            WAIT_V(6); BAR; MMA(1, 1, At, B1); BAR;
            a1 = a2 + KTB;
        }
        if (nt_done < 6) STAMP(6 + 4 * nt_done);
        unsigned tz = (unsigned)tid; asm volatile("" : "+v"(tz));
        const unsigned svo = tz * 16u;
        const __amdgpu_buffer_rsrc_t d_sl = mk_rsrc(P.ws[rank] + OFF_SLAB + (size_t)IT_S(cur) * SLAB, SLAB);
        unsigned* sflag = ctl + C_SLAB + IT_S(cur) * 8 + wid;
        if (SLAB16 && kind == 3) {
#pragma unroll
            for (int k = 0; k < 16; ++k) {
                const f32x4 v0 = acc[k >> 3][k & 1][(k >> 1) & 3][0], v1 = acc[k >> 3][k & 1][(k >> 1) & 3][1];
                u32x4 wv; wv.x = cvt_pk_bf16(v0[0], v0[1]); wv.y = cvt_pk_bf16(v0[2], v0[3]); wv.z = cvt_pk_bf16(v1[0], v1[1]); wv.w = cvt_pk_bf16(v1[2], v1[3]);
                if (bad) wv = (u32x4){0x7fc07fc0u, 0x7fc07fc0u, 0x7fc07fc0u, 0x7fc07fc0u};
                ST128(wv, d_sl, svo, (unsigned)k * 8192u, AUX_AGT);
            }
            WAIT_V(0);
            if (lane == 0) st_agt(sflag, call);
        } else if (kind == 1 || kind == 3) {
#pragma unroll
            for (int k = 0; k < 32; ++k) {
                u32x4 sv = __builtin_bit_cast(u32x4, acc[k >> 4][(k >> 1) & 1][(k >> 2) & 3][k & 1]);
                if (bad) sv = (u32x4){0x7fc00000u, 0x7fc00000u, 0x7fc00000u, 0x7fc00000u};
                ST128(sv, d_sl, svo, (unsigned)k * 8192u, AUX_AGT);
            }
            WAIT_V(0);
            if (lane == 0) st_agt(sflag, call);
        } else {
            const __amdgpu_buffer_rsrc_t d_out = mk_rsrc(P.out[rank], OUT_BYTES);
            const unsigned obase = (unsigned)(wr * 64 + (tz & 15u)) * OUT_PITCH + (unsigned)(wc * 32 + 8 * ((tz >> 4) & 3u)) * 2u;
            const unsigned sbase = (unsigned)(256 * IT_I(cur)) * OUT_PITCH + (unsigned)wpanel(IT_B(cur), rank) * 512u;
#define PUT(ai, m, bj, v0, v1) do { u32x4 wv; wv.x = cvt_pk_bf16(v0[0], v0[1]); wv.y = cvt_pk_bf16(v0[2], v0[3]); wv.z = cvt_pk_bf16(v1[0], v1[1]); wv.w = cvt_pk_bf16(v1[2], v1[3]); \
                if (bad) wv = (u32x4){0x7fc07fc0u, 0x7fc07fc0u, 0x7fc07fc0u, 0x7fc07fc0u}; \
                ST128(wv, d_out, obase + (bj) * 256u, sbase + (unsigned)((ai) * 128 + (m) * 16) * OUT_PITCH, AUX_SYS); } while (0)
            if (kind != 2) {
#pragma unroll
                for (int ai = 0; ai < 2; ++ai)
#pragma unroll
                    for (int m = 0; m < 4; ++m)
#pragma unroll
                        for (int bj = 0; bj < 2; ++bj) { const f32x4 v0 = acc[ai][bj][m][0], v1 = acc[ai][bj][m][1]; PUT(ai, m, bj, v0, v1); }
            } else {
                const u64 t0 = __builtin_amdgcn_s_memrealtime();
                for (;;) {
                    const unsigned f = (unsigned)RFL(ld_agt(sflag));
                    if (f >= call) break;
                    if (expired(ctl, call, t0, lane)) { bad = 1u; break; }
                    __builtin_amdgcn_s_sleep(4);
                }
#pragma unroll
                for (int g8 = 0; g8 < 4; ++g8) {
                    u32x4 pv[8];
#pragma unroll
                    for (int k = 0; k < 8; ++k) pv[k] = __builtin_amdgcn_raw_buffer_load_b128(d_sl, svo, (unsigned)(g8 * 8 + k) * 8192u, AUX_AGT);
#pragma unroll
                    for (int h = 0; h < 4; ++h) {
                        const int ai = g8 >> 1, m = (g8 & 1) * 2 + (h >> 1), bj = h & 1;
                        const f32x4 v0 = acc[ai][bj][m][0] + __builtin_bit_cast(f32x4, pv[2 * h]), v1 = acc[ai][bj][m][1] + __builtin_bit_cast(f32x4, pv[2 * h + 1]);
                        PUT(ai, m, bj, v0, v1);
                    }
                    SCHED;
                }
            }
#undef PUT
            if (IT_B(cur) < NSH) {
                WAIT_V(0);
                unsigned c = 0;
                if (lane == 0) c = __hip_atomic_fetch_add(mbox + 4, 1u, __ATOMIC_RELAXED, __HIP_MEMORY_SCOPE_WORKGROUP);
                if (((unsigned)RFL(c) & 7u) == 7u) signal_ranks(P, 0xffu & ~(1u << rank), C_YF + rank * 128 + IT_Y(cur), call, lane);
            }
        }
        if (nt_done < 6) STAMP(7 + 4 * nt_done);
        ++nt_done;
        if (nxt < 0) break;
        if (IT_K(nxt) == 4) {
            const __amdgpu_buffer_rsrc_t s_sl = mk_rsrc(P.ws[rank] + OFF_SLAB + (size_t)IT_S(nxt) * SLAB, SLAB);
            u32x4 pv[16];
#pragma unroll
            for (int k = 0; k < 16; ++k) pv[k] = __builtin_amdgcn_raw_buffer_load_b128(s_sl, svo, (unsigned)k * 8192u, AUX_AGT);
#pragma unroll
            for (int k = 0; k < 16; ++k) {
                acc[k >> 3][k & 1][(k >> 1) & 3][0] = __builtin_bit_cast(f32x4, (u32x4){pv[k].x << 16, pv[k].x & 0xffff0000u, pv[k].y << 16, pv[k].y & 0xffff0000u});
                acc[k >> 3][k & 1][(k >> 1) & 3][1] = __builtin_bit_cast(f32x4, (u32x4){pv[k].z << 16, pv[k].z & 0xffff0000u, pv[k].w << 16, pv[k].w & 0xffff0000u});
            }
#pragma unroll
            for (int k = 0; k < 32; k += 4)
                asm volatile("" : "+v"(acc[k >> 4][0][(k >> 2) & 3][0]), "+v"(acc[k >> 4][0][(k >> 2) & 3][1]), "+v"(acc[k >> 4][1][(k >> 2) & 3][0]), "+v"(acc[k >> 4][1][(k >> 2) & 3][1]));
        } else ZERO_ACC();
        bad = nbad;
        dyn = dyn || IT_K(nxt) != 3;
        cur = nxt; cA = nA; cB = nB;
    }
    WAIT_V(0);
    if (wr == 0) BAR;
#undef SA
#undef SB
#undef STAGE
#undef LDA
#undef LDB
#undef MMA
#undef ZERO_ACC
}

__global__ __launch_bounds__(512, 2) void fused_gemm_ar(Params P) {
    extern __shared__ __attribute__((aligned(16))) unsigned char shm[];
    LAS unsigned char* lds = (LAS unsigned char*)shm;
    const int b = blockIdx.x, rank = P.rank;
    unsigned* ctl = (unsigned*)P.ws[rank];
    const unsigned call = *(const CAS unsigned*)(u64)(ctl + C_FIN) / (unsigned)GRID + 1u;
    STAMP(0);
    if (b < NPULL) {
        puller(P, lds, b, call);
    } else {
        const int g = b - NPULL;
        stager(P, lds, g, call);
        STAMP(2);
        gemm_worker(P, lds, g, call);
    }
    __syncthreads();
    STAMP(1);
    STAMP(60 + (call & 1));
    if (threadIdx.x == 0) __hip_atomic_fetch_add(ctl + C_FIN, 1u, __ATOMIC_RELAXED, __HIP_MEMORY_SCOPE_AGENT);
}
}

extern "C" void kernel_launch(int rank, int nranks, ncclComm_t comm,
                              hipStream_t stream,
                              void* const* d_in, const int* in_sizes, int n_in,
                              void* d_out, int out_size,
                              const int* in_dtypes, int out_dtype,
                              void* const* d_out_all,
                              void* const* d_ws_all, size_t ws_bytes) {
    if (nranks != NR || (unsigned)rank >= (unsigned)NR || ws_bytes < WS_USED || n_in < 2 ||
        in_sizes[0] != MM * KS || in_sizes[1] != KS * NN || out_size != MM * NN) {
        fprintf(stderr, "kernel_launch: unexpected problem shape\n");
        return;
    }
    static unsigned attr_done = 0;
    int dev = 0; HIP_CHECK(hipGetDevice(&dev));
    if (!((attr_done >> (dev & 31)) & 1u)) {
        HIP_CHECK(hipFuncSetAttribute((const void*)fused_gemm_ar, hipFuncAttributeMaxDynamicSharedMemorySize, (int)LDS_BYTES));
        attr_done |= 1u << (dev & 31);
    }
    Params P;
    P.x = (const unsigned short*)d_in[0];
    P.w = (const unsigned short*)d_in[1];
    for (int p = 0; p < NR; ++p) { P.ws[p] = (char*)d_ws_all[p]; P.out[p] = (char*)d_out_all[p]; }
    P.rank = rank;
    hipLaunchKernelGGL(fused_gemm_ar, dim3(GRID), dim3(512), LDS_BYTES, stream, P);
    HIP_CHECK(hipGetLastError());
}
```

```cpp
#include <hip/hip_runtime.h>
#include <rccl/rccl.h>
#include <cstdio>
#include <cstdint>

#define HIP_CHECK(expr) do { hipError_t _e = (expr); if (_e != hipSuccess) { \
    fprintf(stderr, "HIP error %s at %s:%d\n", hipGetErrorString(_e), __FILE__, __LINE__); \
    return; } } while (0)

#define EREP 4
#define PUSH0 1
#define RTAIL 6
#define NPRE 48
#define PRIO1 40
#define PRIO2 112
#define DLY1 150
#define DLY2 300
#define SADDR 1
#define SLAB16 1
#define PPG 6
#define WPL (PPG * 64 / 7)
#define KTICKS 100

namespace {
#define LAS __attribute__((address_space(3)))
#define GAS __attribute__((address_space(1)))
#define CAS __attribute__((address_space(4)))
typedef short bf16x8 __attribute__((ext_vector_type(8)));
typedef float f32x4 __attribute__((ext_vector_type(4)));
typedef unsigned u32x4 __attribute__((ext_vector_type(4)));
typedef unsigned u32x2 __attribute__((ext_vector_type(2)));
typedef unsigned long long u64;

constexpr int NR = 8;
constexpr int MM = 4096, NN = 16384, KS = 512;
constexpr unsigned IMG = 16384;
constexpr unsigned KTB = 2 * IMG;
constexpr unsigned CHUNK = 8 * KTB;
constexpr unsigned PANEL = NR * CHUNK;
constexpr int NXP = MM / 256, NWP = NN / 256;
constexpr int WPR = NWP / NR;
constexpr int NSH = WPR - EREP;
constexpr int NW = NSH + NR * EREP;
constexpr unsigned OFF_X = 1u << 20;
constexpr unsigned OFF_W = OFF_X + NXP * PANEL;
constexpr unsigned PAR_STRIDE = (NXP + NWP) * PANEL;
constexpr unsigned OFF_SLAB = OFF_X + 2 * PAR_STRIDE;
constexpr unsigned SLAB = 256 * 256 * 4;
constexpr unsigned WS_USED = OFF_SLAB + 512 * SLAB;
constexpr unsigned OUT_BYTES = (unsigned)MM * NN * 2;
constexpr unsigned OUT_PITCH = NN * 2;
constexpr int NCHUNK = NXP + NWP;
constexpr int XPUSH = PUSH0 ? 1 : 0;
constexpr int NSEQ = NXP + NW - XPUSH;
constexpr int PRE_COST = 150, MAXSLAB = 512;
constexpr int NYT = NXP * NSH;
constexpr int NIN_IT = NSEQ * 64;
constexpr int NTOT_IT = NIN_IT + NYT * 64;
constexpr int GRID = 256, NPULL = 8 * PPG, NWORK = GRID - NPULL, WPG = NWORK / 8;
constexpr int NJOB = NCHUNK * 4;
constexpr int C_FIN = 32, C_TMO = 64, C_SDONE = 96;
constexpr int C_XP = 128  ;
constexpr int C_GQ = 448, C_TQ = 768  ;
constexpr int C_Q = 5120  ;
constexpr int C_STG = 1024  ;
constexpr int C_YF = 8192  ;
constexpr int C_ARR = 12288  ;
constexpr int C_SLAB = 40960  ;
constexpr int C_ACNT = 16384  ;
constexpr unsigned LDS_BYTES = 8 * IMG + 256;
constexpr u64 DEADLINE = 40000000ull;
constexpr int AUX_SYS = 17;
constexpr int AUX_AGT = 16;
static_assert(RTAIL <= 16 && NSEQ <= 64 && NYT <= 128 && C_ACNT + 64 * 8 * 32 <= C_SLAB && (C_SLAB + 512 * 8) * 4 <= (512 << 10) && PPG >= 4, "layout");

struct Params {
    const unsigned short* x;
    const unsigned short* w;
    char* ws[NR];
    char* out[NR];
    int rank;
};

constexpr int pos_x(int i) { return 2 * i; }
constexpr int pos_w(int b) { return b < NXP ? 2 * b + 1 : b + NXP; }
constexpr int tile_pos(int i, int b) { return pos_x(i) > pos_w(b) ? pos_x(i) : pos_w(b); }
constexpr int NSEQ_POS = (NW >= NXP) ? NXP + NW : 2 * NXP;
#define IT_I(e) ((e) & 15)
#define IT_B(e) (((e) >> 4) & 63)
#define IT_Y(e) (((e) >> 10) & 127)
#define IT_K(e) (((e) >> 17) & 7)
#define IT_S(e) ((e) >> 20)
constexpr int MAXPRE = 4;
struct Tables {
    int tile[8][96];
    int ntile[8];
    int pre[8][WPG][MAXPRE + 1];
    int gtile[2 * 8 * 16];
    int ngt, nslab;
    int ytile[128];
    int seq[64];
};
constexpr Tables make_tables() {
    Tables T{};
    int ny = 0, ns = 0;
    for (int pos = 0; pos < NSEQ_POS; ++pos) {
        for (int i = XPUSH; i < NXP; ++i) if (pos_x(i) == pos) T.seq[ns++] = i;
        for (int b = 0; b < NW; ++b) if (pos_w(b) == pos) T.seq[ns++] = -1 - b;
        for (int b = 0; b < NW; ++b)
            for (int i = 0; i < NXP; ++i)
                if (tile_pos(i, b) == pos) {
                    const int q = (i & 3) + 4 * (b & 1);
                    T.tile[q][T.ntile[q]++] = i | (b << 4) | (b < NSH ? ny << 10 : 0);
                    if (b < NSH) T.ytile[ny++] = i | (b << 4);
                }
    }
    int gt[8 * 16] = {}; int ng = 0;
    for (int k = RTAIL; k >= 1; --k)
        for (int q = 0; q < 8; ++q)
            if (T.ntile[q] - RTAIL >= WPG) gt[ng++] = T.tile[q][T.ntile[q] - k];
    for (int q = 0; q < 8; ++q)
        if (T.ntile[q] - RTAIL >= WPG) T.ntile[q] -= RTAIL;
    constexpr int LAG = 8;
    for (int k = 0; k < ng + LAG; ++k) {
        if (k < ng) T.gtile[T.ngt++] = gt[k] | (1 << 17) | (k << 20);
        if (k >= LAG) T.gtile[T.ngt++] = gt[k - LAG] | (2 << 17) | ((k - LAG) << 20);
    }
    T.nslab = ng;
    for (int q = 0; q < 8; ++q) {
        int np[WPG] = {}, slack[WPG] = {};
        for (int w = 0; w < WPG; ++w) {
            for (int k = 0; k <= MAXPRE; ++k) T.pre[q][w][k] = -1;
            slack[w] = w < T.ntile[q] ? 60 + 44 * (tile_pos(IT_I(T.tile[q][w]), IT_B(T.tile[q][w])) + 1 - XPUSH) - 240 : 0;
        }
        for (int k = WPG; k < T.ntile[q] && k < WPG + NPRE; ++k) {
            int best = 0;
            for (int w = 1; w < WPG; ++w) if (slack[w] > slack[best]) best = w;
            if (slack[best] < PRE_COST || np[best] >= MAXPRE) break;
            T.pre[q][best][np[best]++] = T.tile[q][k] | (3 << 17) | (T.nslab << 20);
            T.tile[q][k] |= (4 << 17) | (T.nslab << 20);
            ++T.nslab; slack[best] -= PRE_COST;
            if (np[best] >= MAXPRE) slack[best] = 0;
        }
    }
    return T;
}
__constant__ const Tables TAB = make_tables();
constexpr int n_active() { const Tables T = make_tables(); int n = 0; for (int q = 0; q < 8; ++q) n += T.ntile[q] < WPG ? T.ntile[q] : WPG; return n; }
constexpr int NACTIVE = n_active();
static_assert(make_tables().nslab <= MAXSLAB, "slabs");

__device__ __forceinline__ int lds_byte(int r, int c) { const int st = (r >> 4) * 2 + (c >> 5), rr = r & 15, cc = c & 31, ob = rr * 64 + cc * 2; return st * 1024 + (ob ^ (((ob >> 9) & 1) << 5)); }
__device__ __forceinline__ void stage_rc(int b, int& R, int& C) { const int st = b / 1024, sb = b % 1024, swz = sb ^ (((sb >> 9) & 1) << 5); R = (st >> 1) * 16 + swz / 64; C = (st & 1) * 32 + (swz % 64) / 2; }
__device__ __forceinline__ int wpanel(int b, int r) { const int m = b - NSH; return b < NSH ? r * WPR + b : (m & 7) * WPR + NSH + (m >> 3); }
__device__ __forceinline__ int seq_chunk(int s, int r) { const int v = TAB.seq[s]; return v >= 0 ? v : NXP + wpanel(-1 - v, r); }
__device__ __forceinline__ unsigned chunk_off(int cid, int src, unsigned call) { return OFF_X + (call & 1u) * PAR_STRIDE + (unsigned)cid * PANEL + (unsigned)src * CHUNK; }

__device__ __forceinline__ __amdgpu_buffer_rsrc_t mk_rsrc(const void* p, unsigned bytes) {
    const u64 a = (u64)p;
    const unsigned lo = __builtin_amdgcn_readfirstlane((unsigned)a), hi = __builtin_amdgcn_readfirstlane((unsigned)(a >> 32));
    return __builtin_amdgcn_make_buffer_rsrc((void*)(((u64)hi << 32) | lo), 0, bytes, 0x00020000);
}
__device__ __forceinline__ unsigned ld_sys(const unsigned* p) { return __hip_atomic_load(p, __ATOMIC_RELAXED, __HIP_MEMORY_SCOPE_SYSTEM); }
__device__ __forceinline__ unsigned ld_agt(const unsigned* p) { return __hip_atomic_load(p, __ATOMIC_RELAXED, __HIP_MEMORY_SCOPE_AGENT); }
__device__ __forceinline__ void st_sys(unsigned* p, unsigned v) { __hip_atomic_store(p, v, __ATOMIC_RELAXED, __HIP_MEMORY_SCOPE_SYSTEM); }
__device__ __forceinline__ void st_agt(unsigned* p, unsigned v) { __hip_atomic_store(p, v, __ATOMIC_RELAXED, __HIP_MEMORY_SCOPE_AGENT); }
#define RFL(x) __builtin_amdgcn_readfirstlane(x)
#define ST128(v, rsrc, voff, soff, aux) __builtin_amdgcn_raw_buffer_store_b128(v, rsrc, (voff) + (soff), 0, aux)
#define STAMP(slot) do { } while (0)
#define WAIT_V(n) asm volatile("s_waitcnt vmcnt(" #n ")" ::: "memory")
#define WAIT_L(n) asm volatile("s_waitcnt lgkmcnt(" #n ")" ::: "memory")
#define BAR __builtin_amdgcn_s_barrier()
#define SCHED __builtin_amdgcn_sched_barrier(0)
#define LANE0_ADD(rsrc, word, lane) (unsigned)__builtin_amdgcn_raw_ptr_buffer_atomic_add_i32(1, rsrc, (lane) == 0 ? (int)((word) * 4) : (int)0xfffffff0u, 0, 0)

__device__ __forceinline__ void signal_ranks(const Params& P, unsigned mask, int word, unsigned val, int lane) {
    const CAS Params* kp = (const CAS Params*)__builtin_amdgcn_kernarg_segment_ptr();
    asm volatile("" : "+s"(kp));
    u64 a = 0;
#pragma unroll
    for (int p = 0; p < NR; ++p) { u64 t = (u64)kp->ws[p]; asm volatile("" : "+s"(t)); a = (lane == p) ? t : a; }
    if (lane < NR && ((mask >> lane) & 1u)) __hip_atomic_store((GAS unsigned*)a + word, val, __ATOMIC_RELAXED, __HIP_MEMORY_SCOPE_SYSTEM);
}
__device__ __forceinline__ bool expired(unsigned* ctl, unsigned call, u64 t0, int lane) {
    const u64 dt = __builtin_amdgcn_s_memrealtime() - t0;
    if (dt < 100000ull) return false;
    const bool dead = dt > DEADLINE || (unsigned)RFL(ld_agt(ctl + C_TMO)) == call;
    if (dead && lane == 0) st_agt(ctl + C_TMO, call);
    return RFL((int)dead) != 0;
}

__device__ __forceinline__ void puller(const Params& P, LAS unsigned char* lds, int b, unsigned call) {
    const int rank = P.rank, lane = threadIdx.x & 63, wid = RFL((int)(threadIdx.x >> 6));
    unsigned* ctl = (unsigned*)P.ws[rank];
    const __amdgpu_buffer_rsrc_t d_ws = mk_rsrc(P.ws[rank], WS_USED), d_out = mk_rsrc(P.out[rank], OUT_BYTES);
    bool dead = false, xp_pend = false;
    LAS unsigned* xcnt = (LAS unsigned*)lds;
    const unsigned xp_peer = (unsigned)(rank + 1 + (b >> 2)) & 7u;
    if (threadIdx.x == 0) *xcnt = 0u;
    __syncthreads();
    if (wid < 4) {
        if (XPUSH && b < 28) {
            const __amdgpu_buffer_rsrc_t s_x = mk_rsrc(P.x, MM * KS * 2), p_ws = mk_rsrc(P.ws[xp_peer], WS_USED);
            const int im = (b & 3) * 4 + wid;
            u32x4 v[16];
#pragma unroll
            for (int q = 0; q < 16; ++q) {
                int R, C; stage_rc(q * 1024 + lane * 16, R, C);
                v[q] = __builtin_amdgcn_raw_buffer_load_b128(s_x, (unsigned)(R * KS + C) * 2u, (unsigned)(128 * (im & 1) * KS + 64 * (im >> 1)) * 2u, 0);
            }
#pragma unroll
            for (int q = 0; q < 16; ++q) ST128(v[q], p_ws, (unsigned)lane * 16u, chunk_off(0, rank, call) + (unsigned)im * IMG + q * 1024u, AUX_SYS);
            xp_pend = true;
        }
    }
    if (b == NPULL - 1 && threadIdx.x < 7) st_agt(ctl + C_Q + 256 * ((call + 1u) & 1u) + 32 * (int)threadIdx.x, 0u);
#define XP_DONE() do { if (XPUSH && xp_pend) { xp_pend = false; \
        if (lane == 0 && __hip_atomic_fetch_add(xcnt, 1u, __ATOMIC_RELAXED, __HIP_MEMORY_SCOPE_WORKGROUP) == 3u) \
            __hip_atomic_fetch_add((unsigned*)P.ws[xp_peer] + C_XP + 32 * rank, 1u, __ATOMIC_RELAXED, __HIP_MEMORY_SCOPE_SYSTEM); } } while (0)
    const int mycid = seq_chunk(lane < NSEQ ? lane : 0, rank);
    const unsigned ytab = (unsigned)TAB.ytile[lane] | ((unsigned)TAB.ytile[64 + lane] << 8);
    const int qw0 = C_Q + 256 * (int)(call & 1u);
    const int u = ((b >> 3) * 8 + wid) * 8 + (b & 7), link = u % 7;
    if (u >= 7 * WPL) { if (xp_pend) { WAIT_V(0); XP_DONE(); } return; }
    {
        const unsigned p = (unsigned)(rank + 1 + link) & 7u;
        const __amdgpu_buffer_rsrc_t s_ws = mk_rsrc(P.ws[p], WS_USED), s_out = mk_rsrc(P.out[p], OUT_BYTES);
        const int qword = qw0 + 32 * link;
        unsigned tk = LANE0_ADD(d_ws, qword, lane);
        unsigned nn = (unsigned)RFL(tk);
        u32x4 vl = {0u, 0u, 0u, 0u}; unsigned ol = 0;
        int pend = -1;
#define FLAG_IF_LAST(c, ps) do { if (((unsigned)RFL(c) & 63u) == 63u && lane == 0) st_agt(ctl + C_ARR + __builtin_amdgcn_readlane(mycid, ps) * 8 + p, call); } while (0)
#define FLUSH_IN() do { ST128(vl, d_ws, ol, 0u, AUX_AGT); WAIT_V(0); const unsigned _c = LANE0_ADD(d_ws, C_ACNT + (pend * 8 + (int)p) * 32, lane); FLAG_IF_LAST(_c, pend); pend = -1; } while (0)
        {
            const unsigned foff = (unsigned)(C_STG + p * 512 + mycid * 4) * 4u;
            u64 rq0 = 0, rq1 = 0, rq2 = 0, rq3 = 0;
            while (nn < (unsigned)NIN_IT) {
                const unsigned n = nn;
                const int s = (int)(n >> 6), qq = (int)(n >> 4) & 3;
                const bool rdy = (((qq == 0 ? rq0 : qq == 1 ? rq1 : qq == 2 ? rq2 : rq3) >> s) & 1) != 0;
                if (!rdy && !dead) {
                    const u64 t0 = __builtin_amdgcn_s_memrealtime();
                    for (;;) {
                        const u32x4 f = __builtin_amdgcn_raw_buffer_load_b128(d_ws, foff, 0, AUX_SYS);
                        rq0 = __ballot(f.x >= call); rq1 = __ballot(f.y >= call); rq2 = __ballot(f.z >= call); rq3 = __ballot(f.w >= call);
                        if (((qq == 0 ? rq0 : qq == 1 ? rq1 : qq == 2 ? rq2 : rq3) >> s) & 1) break;
                        if (xp_pend) { WAIT_V(0); XP_DONE(); }
                        if (pend >= 0) FLUSH_IN();
                        if (expired(ctl, call, t0, lane)) { dead = true; break; }
                        __builtin_amdgcn_s_sleep(1);
                    }
                }
                const int cid = __builtin_amdgcn_readlane(mycid, s);
                const unsigned soff = chunk_off(cid, (int)p, call) + (n & 63u) * 4096u, voff = (unsigned)lane * 16u;
                const unsigned so = dead ? 0xfffff000u : soff;
                tk = LANE0_ADD(d_ws, qword, lane);
                const u32x4 v0 = __builtin_amdgcn_raw_buffer_load_b128(s_ws, voff, so, AUX_SYS);
                ST128(vl, d_ws, pend >= 0 ? ol : 0xfffffff0u, 0u, AUX_AGT);
                WAIT_V(1);
                const u32x4 v1 = __builtin_amdgcn_raw_buffer_load_b128(s_ws, voff + 1024u, so, AUX_SYS);
                ST128(v0, d_ws, voff, soff, AUX_AGT);
                nn = (unsigned)RFL(tk);
                WAIT_V(1);
                XP_DONE();
                const unsigned c = LANE0_ADD(d_ws, pend >= 0 ? C_ACNT + (pend * 8 + (int)p) * 32 : 0x3ffffff0, lane);
                const u32x4 v2 = __builtin_amdgcn_raw_buffer_load_b128(s_ws, voff + 2048u, so, AUX_SYS);
                ST128(v1, d_ws, voff, soff + 1024u, AUX_AGT);
                WAIT_V(1);
                if (pend >= 0) FLAG_IF_LAST(c, pend);
                const u32x4 v3 = __builtin_amdgcn_raw_buffer_load_b128(s_ws, voff + 3072u, so, AUX_SYS);
                ST128(v2, d_ws, voff, soff + 2048u, AUX_AGT);
                vl = v3; ol = voff + soff + 3072u; pend = s;
                asm volatile("" : "+v"(vl));
            }
            if (pend >= 0) FLUSH_IN();
            if (xp_pend) { WAIT_V(0); XP_DONE(); }
        }
        {
            const unsigned* fl = ctl + C_YF + p * 128 + lane;
            const unsigned voff = (unsigned)(lane >> 5) * OUT_PITCH + p * (WPR * 512u) + (unsigned)(lane & 31) * 16u;
            u64 r0 = 0, r1 = 0;
            ol = 0xfffffff0u;
            while (nn < (unsigned)NTOT_IT) {
                const unsigned n = nn - (unsigned)NIN_IT;
                const int o = (int)(n >> 6);
                if (!dead && !(((o < 64 ? r0 : r1) >> (o & 63)) & 1)) {
                    const u64 t0 = __builtin_amdgcn_s_memrealtime();
                    for (;;) {
                        const unsigned f0 = ld_sys(fl), f1 = ld_sys(fl + 64);
                        r0 = __ballot(f0 >= call); r1 = __ballot(f1 >= call);
                        if (((o < 64 ? r0 : r1) >> (o & 63)) & 1) break;
                        if (expired(ctl, call, t0, lane)) { dead = true; break; }
                        __builtin_amdgcn_s_sleep(1);
                    }
                }
                const unsigned e = ((unsigned)__builtin_amdgcn_readlane((int)ytab, o & 63) >> ((o >> 6) * 8)) & 255u;
                const unsigned soff = ((e & 15u) * 256u + (n & 63u) * 4u) * OUT_PITCH + (e >> 4) * 512u;
                const int o2 = o + 3 < NYT ? o + 3 : NYT - 1;
                const bool refresh = !(((o2 < 64 ? r0 : r1) >> (o2 & 63)) & 1);
                unsigned f0 = 0, f1 = 0;
                if (refresh) { f0 = ld_sys(fl); f1 = ld_sys(fl + 64); }
                tk = LANE0_ADD(d_ws, qword, lane);
                u32x4 v0 = __builtin_amdgcn_raw_buffer_load_b128(s_out, voff, soff, AUX_SYS);
                ST128(vl, d_out, ol, 0u, AUX_AGT);
                WAIT_V(1);
                u32x4 v1 = __builtin_amdgcn_raw_buffer_load_b128(s_out, voff, soff + 2 * OUT_PITCH, AUX_SYS);
                if (dead) v0 = (u32x4){0x7fc07fc0u, 0x7fc07fc0u, 0x7fc07fc0u, 0x7fc07fc0u};
                ST128(v0, d_out, voff, soff, AUX_AGT);
                nn = (unsigned)RFL(tk);
                if (refresh) { r0 = __ballot(f0 >= call); r1 = __ballot(f1 >= call); }
                if (dead) v1 = (u32x4){0x7fc07fc0u, 0x7fc07fc0u, 0x7fc07fc0u, 0x7fc07fc0u};
                vl = v1; ol = voff + soff + 2 * OUT_PITCH;
                asm volatile("" : "+v"(vl));
            }
            ST128(vl, d_out, ol, 0u, AUX_AGT);
        }
    }
#undef FLUSH_IN
#undef FLAG_IF_LAST
#undef XP_DONE
}

__device__ __forceinline__ void job_of(int idx, int& cid, int& q) {
    q = idx & 3;
    int u = idx >> 2;
    if (u < 9 * NSH) { const int g = u / 9, r = u % 9; cid = r < 8 ? NXP + r * WPR + g : (g == 0 ? XPUSH : g == 1 ? 1 - XPUSH : g); return; }
    u -= 9 * NSH;
    constexpr int NXR = NXP - NSH, NPAIR = NXR < 8 * EREP ? NXR : 8 * EREP;
    if (u < 2 * NPAIR) { cid = (u & 1) ? NXP + wpanel(NSH + (u >> 1), 0) : NSH + (u >> 1); return; }
    u -= 2 * NPAIR;
    cid = NXR > NPAIR ? NSH + NPAIR + u : NXP + wpanel(NSH + NPAIR + u, 0);
}
__device__ __forceinline__ void stager(const Params& P, LAS unsigned char* lds, int g, unsigned call) {
    const int rank = P.rank, tid = threadIdx.x, lane = tid & 63;
    const __amdgpu_buffer_rsrc_t d_ws = mk_rsrc(P.ws[rank], WS_USED);
    const __amdgpu_buffer_rsrc_t s_x = mk_rsrc(P.x, MM * KS * 2), s_w = mk_rsrc(P.w, KS * NN * 2);
    if (g >= PRIO1) {
        const u64 t0 = __builtin_amdgcn_s_memrealtime(), dly = g >= PRIO2 ? DLY2 : DLY1;
        while (__builtin_amdgcn_s_memrealtime() - t0 < dly) __builtin_amdgcn_s_sleep(4);
    }
    for (int idx = g; idx < NJOB; idx += NWORK) {
        int cid, q; job_of(idx, cid, q);
        const unsigned dst = chunk_off(cid, rank, call) + (unsigned)q * (2 * KTB) + (unsigned)tid * 16u;
        if (cid < NXP) {
            u32x4 v[8];
#pragma unroll
            for (int u = 0; u < 8; ++u) {
                int R, C; stage_rc((u & 1) * 8192 + tid * 16, R, C);
                v[u] = __builtin_amdgcn_raw_buffer_load_b128(s_x, (unsigned)(R * KS + C) * 2u, (unsigned)((256 * cid + 128 * ((u >> 1) & 1)) * KS + 128 * q + 64 * (u >> 2)) * 2u, 0);
            }
#pragma unroll
            for (int u = 0; u < 8; ++u) ST128(v[u], d_ws, dst + u * 8192u, 0, AUX_SYS);
        } else {
            const int jg = cid - NXP, ng = tid & 31, kg = tid >> 5;
            u32x4 v[8];
#pragma unroll
            for (int r = 0; r < 8; ++r) v[r] = __builtin_amdgcn_raw_buffer_load_b128(s_w, (unsigned)(8 * kg * NN + 8 * ng) * 2u, (unsigned)((128 * q + r) * NN + 256 * jg) * 2u, 0);
            const int lbase = (kg >> 3) * (int)KTB + (ng >> 4) * (int)IMG;
#pragma unroll
            for (int jj = 0; jj < 8; ++jj) {
                u32x4 o;
#pragma unroll
                for (int d = 0; d < 4; ++d)
                    o[d] = (jj & 1) ? __builtin_amdgcn_perm(v[2 * d + 1][jj >> 1], v[2 * d][jj >> 1], 0x07060302u)
                                    : __builtin_amdgcn_perm(v[2 * d + 1][jj >> 1], v[2 * d][jj >> 1], 0x05040100u);
                const int R = 32 * ((ng & 15) >> 2) + 16 * (jj >> 2) + 4 * (ng & 3) + (jj & 3);
                *(LAS u32x4*)(lds + lbase + lds_byte(R, (kg & 7) * 8)) = o;
            }
            __syncthreads();
#pragma unroll
            for (int u = 0; u < 8; ++u) v[u] = *(const LAS u32x4*)(lds + u * 8192 + tid * 16);
#pragma unroll
            for (int u = 0; u < 8; ++u) ST128(v[u], d_ws, dst + u * 8192u, 0, AUX_SYS);
        }
        WAIT_V(0);
        __syncthreads();
        const unsigned mask = cid < XPUSH ? (1u << rank) : (cid < NXP || ((cid - NXP) & 7) >= NSH) ? 0xffu : (1u << ((cid - NXP) >> 3));
        if (tid < 64) signal_ranks(P, mask, C_STG + rank * 512 + cid * 4 + q, call, lane);
        if (idx == g) STAMP(30);
    }
    if (tid == 0) __hip_atomic_fetch_add((unsigned*)P.ws[rank] + C_SDONE, 1u, __ATOMIC_RELAXED, __HIP_MEMORY_SCOPE_AGENT);
}

__device__ __forceinline__ unsigned cvt_pk_bf16(float lo, float hi) { unsigned r; asm volatile("v_cvt_pk_bf16_f32 %0, %1, %2" : "=v"(r) : "v"(lo), "v"(hi)); return r; }

__device__ __forceinline__ unsigned wait_inputs(unsigned* ctl, unsigned call, int rank, int lane, bool arr, int i, int jg, bool sd, int slab) {
    const bool xp = i < XPUSH;
    const unsigned* w0 = lane >= 24 ? ctl + C_SLAB + (slab < 0 ? 0 : slab) * 8 + (lane & 7)
                       : lane == 16 ? ctl + C_SDONE : lane == 17 ? ctl + C_TMO
                       : (xp && lane < 8) ? ctl + C_XP + 32 * lane : ctl + C_ARR + ((lane & 8) ? NXP + jg : i) * 8 + (lane & 7);
    const unsigned tgt = lane == 16 ? (unsigned)NWORK * call : (xp && lane < 8) ? 4u * call : call;
    const bool act = lane < 16 ? (arr && (lane & 7) != rank) : lane == 16 ? sd : (lane >= 24 && lane < 32 && slab >= 0);
    const u64 t0 = __builtin_amdgcn_s_memrealtime();
    unsigned bad = 0;
    for (;;) {
        const unsigned f = (act || lane == 17) ? ld_sys(w0) : tgt;
        if ((unsigned)__builtin_amdgcn_readlane((int)f, 17) == call) { bad = 1u; break; }
        if (!__ballot(act && (int)(f - tgt) < 0)) break;
        if (expired(ctl, call, t0, lane)) { bad = 1u; break; }
        __builtin_amdgcn_s_sleep(8);
    }
    if (arr && xp) { __builtin_amdgcn_fence(__ATOMIC_ACQUIRE, ""); WAIT_V(0); }
    return bad;
}

__device__ __forceinline__ void gemm_worker(const Params& P, LAS unsigned char* lds, int g, unsigned call) {
    const int rank = P.rank, grp = g & 7;
    const int ntq = TAB.ntile[grp];
    if ((g >> 3) >= ntq) return;
    const int tid = threadIdx.x, wid = RFL(tid >> 6), lane = tid & 63, wr = wid >> 2, wc = wid & 3, fr = lane & 15, fq = lane >> 4;
    unsigned* ctl = (unsigned*)P.ws[rank];
    LAS unsigned* mbox = (LAS unsigned*)(lds + 8 * IMG);
    const char* wsb = P.ws[rank] + (size_t)((call & 1u) * PAR_STRIDE);
    const unsigned voff0 = (unsigned)tid * 16u, voff1 = voff0 + 8192u;
    const unsigned ldsw = (unsigned)wid * 1024u;
    const int aoff = lds_byte(wr * 64 + fr, fq * 8), boff = lds_byte(wc * 32 + fr, fq * 8);
    const int own8 = rank * 8;
    const unsigned ownoff = (unsigned)own8 * KTB;
#define SA(b, h) (((b) * 2 + (h)) * IMG)
#define SB(b, h) ((4 + (b) * 2 + (h)) * IMG)
    const __amdgpu_buffer_rsrc_t s_pan = mk_rsrc(wsb, OFF_X + PAR_STRIDE);
#define STAGE(bufoff, gbase) do { \
        __builtin_amdgcn_raw_ptr_buffer_load_lds(s_pan, (LAS void*)(lds + (bufoff) + ldsw), 16, voff0, (gbase), 0, 0); \
        __builtin_amdgcn_raw_ptr_buffer_load_lds(s_pan, (LAS void*)(lds + (bufoff) + ldsw + 8192), 16, voff1, (gbase), 0, 0); } while (0)
#define LDA(dst, b, h) do { _Pragma("unroll") for (int m = 0; m < 4; ++m) _Pragma("unroll") for (int k = 0; k < 2; ++k) dst[m][k] = *(const LAS bf16x8*)(lds + SA(b, h) + aoff + m * 2048 + k * 1024); } while (0)
#define LDB(dst, b, h) do { _Pragma("unroll") for (int n = 0; n < 2; ++n) _Pragma("unroll") for (int k = 0; k < 2; ++k) dst[n][k] = *(const LAS bf16x8*)(lds + SB(b, h) + boff + n * 2048 + k * 1024); } while (0)
#define MMA(ai, bj, At, Bt) do { __builtin_amdgcn_s_setprio(1); _Pragma("unroll") for (int m = 0; m < 4; ++m) _Pragma("unroll") for (int n = 0; n < 2; ++n) _Pragma("unroll") for (int k = 0; k < 2; ++k) \
        acc[ai][bj][m][n] = __builtin_amdgcn_mfma_f32_16x16x32_bf16(Bt[n][k], At[m][k], acc[ai][bj][m][n], 0, 0, 0); __builtin_amdgcn_s_setprio(0); } while (0)
#define ZERO_ACC() do { _Pragma("unroll") for (int a = 0; a < 2; ++a) _Pragma("unroll") for (int b = 0; b < 2; ++b) _Pragma("unroll") for (int m = 0; m < 4; ++m) _Pragma("unroll") for (int n = 0; n < 2; ++n) \
        acc[a][b][m][n] = (f32x4){0.f, 0.f, 0.f, 0.f}; } while (0)
    f32x4 acc[2][2][4][2];
    ZERO_ACC();
    bf16x8 At[4][2], B0[2][2], B1[2][2];
    const int slot = g >> 3;
    int sidx = 0;
    int cur = TAB.pre[grp][slot][0];
    if (cur < 0) cur = TAB.tile[grp][slot];
    bool dyn = IT_K(cur) != 3;
    unsigned cA = OFF_X + (unsigned)IT_I(cur) * PANEL, cB = OFF_W + (unsigned)wpanel(IT_B(cur), rank) * PANEL;
    if (wid == 0) {
        if (lane == 0) mbox[4] = 0u;
        const u64 t0 = __builtin_amdgcn_s_memrealtime();
        const unsigned* w0 = ctl + C_STG + rank * 512 + ((lane & 4) ? NXP + wpanel(IT_B(cur), rank) : IT_I(cur)) * 4 + (lane & 3);
        for (;;) {
            const unsigned f = lane < 8 ? ld_agt(w0) : call;
            if (__all(f >= call)) break;
            if (expired(ctl, call, t0, lane)) break;
            __builtin_amdgcn_s_sleep(4);
        }
        __builtin_amdgcn_fence(__ATOMIC_ACQUIRE, "agent");
        WAIT_V(0); WAIT_L(0);
    }
    asm volatile("" ::: "memory"); BAR; asm volatile("" ::: "memory");
    STAMP(3);
    STAGE(SB(0, 0), cB + ownoff); STAGE(SA(0, 0), cA + ownoff); STAGE(SB(0, 1), cB + ownoff + IMG); STAGE(SA(0, 1), cA + ownoff + IMG);
    if (wr == 1) BAR;
    WAIT_V(4); BAR;
    STAGE(SB(1, 0), cB + ownoff + KTB); STAGE(SA(1, 0), cA + ownoff + KTB); STAGE(SB(1, 1), cB + ownoff + KTB + IMG);
    WAIT_V(6); BAR;
    unsigned a1 = cA + ownoff + KTB;
    int nt_done = 0, rho = 0;
    unsigned bad = 0;
    bool use_gq = false, drew = false;
    for (;;) {
        const int kind = IT_K(cur);
        const bool half = kind == 1 || kind == 2;
        const int nown = kind == 4 ? 0 : half ? 4 : 8, nkt = nown + (kind == 3 ? 0 : half ? 28 : 56);
        const int th = (kind == 0 || kind == 3) ? 6 : 2, rlo = kind == 2 ? 28 : 0, rhi = kind == 1 ? 28 : 56;
        const unsigned own0 = ownoff + (kind == 2 ? 4u * KTB : 0u);
        int nxt = -1; unsigned nbad = 0;
        unsigned nA = cA, nB = cB;
        for (int t = 0; t < nkt; t += 2) {
            const bool hookA = t == th, hookB = t == nkt - 2 && nxt >= 0 && IT_K(nxt) == 4;
            if (hookA && wid == 0) {
                const __amdgpu_buffer_rsrc_t d_ws = mk_rsrc(ctl, 1u << 20);
                const int qw = use_gq ? C_GQ : C_TQ + 32 * grp;
                unsigned tk = 0;
                if (dyn && drew) tk = LANE0_ADD(d_ws, qw, lane);
                if (nt_done < 6) STAMP(4 + 4 * nt_done);
                unsigned tm = 0;
                if (kind != 4) tm = wait_inputs(ctl, call, rank, lane, kind != 3, IT_I(cur), wpanel(IT_B(cur), rank), !drew, -1);
                int e = -1;
                if (!dyn) {
                    e = TAB.pre[grp][slot][++sidx];
                    if (e < 0) e = TAB.tile[grp][slot];
                } else {
                    if (!drew) { tk = LANE0_ADD(d_ws, qw, lane); drew = true; }
                    if (!use_gq) {
                        const unsigned nx = (unsigned)WPG + (unsigned)RFL(tk) - (call - 1u) * (unsigned)ntq;
                        if (nx < (unsigned)ntq) e = TAB.tile[grp][nx];
                        else { use_gq = true; tk = LANE0_ADD(d_ws, C_GQ, lane); }
                    }
                    if (use_gq) {
                        const unsigned nx = (unsigned)RFL(tk) - (call - 1u) * (unsigned)(TAB.ngt + NACTIVE);
                        if (nx < (unsigned)TAB.ngt) e = TAB.gtile[nx];
                    }
                }
                asm volatile("" :: "v"(tk));
                if (lane == 0) {
                    const unsigned ph = (unsigned)((__builtin_amdgcn_s_memrealtime() / (2 * KTICKS)) % 28ull);
                    mbox[0] = half ? (unsigned)rlo + (ph % 14u) * 2u : ph * 2u;
                    mbox[1] = (unsigned)e;
                    mbox[2] = tm;
                }
                if (nt_done < 6) STAMP(5 + 4 * nt_done);
            }
            if (hookB && wid == 0) {
                const unsigned tm = wait_inputs(ctl, call, rank, lane, true, IT_I(nxt), wpanel(IT_B(nxt), rank), false, IT_S(nxt));
                if (lane == 0) {
                    mbox[0] = (unsigned)((__builtin_amdgcn_s_memrealtime() / (2 * KTICKS)) % 28ull) * 2u;
                    mbox[2] = tm;
                }
            }
            LDB(B0, 0, 0); SCHED; LDA(At, 0, 0); STAGE(SA(1, 1), a1 + IMG);
            WAIT_L(8); BAR;
            u32x4 mb = {0u, 0u, 0u, 0u};
            if (hookA || hookB) mb = *(volatile LAS u32x4*)mbox;
            WAIT_L(0); MMA(0, 0, At, B0); BAR; SCHED;
            if (hookA) {
                if (kind != 4) { rho = RFL((int)mb.x); bad = (unsigned)RFL((int)mb.z); }
                nxt = RFL((int)mb.y);
                if (nxt >= 0) { nA = OFF_X + (unsigned)IT_I(nxt) * PANEL; nB = OFF_W + (unsigned)wpanel(IT_B(nxt), rank) * PANEL; }
            }
            unsigned a2, b2;
            if (t + 2 < nown) { a2 = cA + own0 + (unsigned)(t + 2) * KTB; b2 = cB + own0 + (unsigned)(t + 2) * KTB; }
            else if (t + 2 < nkt) { const unsigned ko = (unsigned)(rho + (rho >= own8 ? 8 : 0)) * KTB; a2 = cA + ko; b2 = cB + ko; rho = rho + 2 == rhi ? rlo : rho + 2; }
            else if (hookB) {
                rho = RFL((int)mb.x); nbad = (unsigned)RFL((int)mb.z);
                const unsigned ko = (unsigned)(rho + (rho >= own8 ? 8 : 0)) * KTB; a2 = nA + ko; b2 = nB + ko; rho = rho == 54 ? 0 : rho + 2;
            } else { const unsigned no = ownoff + ((nxt >= 0 && IT_K(nxt) == 2) ? 4u * KTB : 0u); a2 = nA + no; b2 = nB + no; }
            LDB(B1, 0, 1); STAGE(SB(0, 0), b2);
            BAR; WAIT_L(0); MMA(0, 1, At, B1); BAR;
            LDA(At, 0, 1); STAGE(SA(0, 0), a2);
            BAR; WAIT_L(0); MMA(1, 0, At, B0); BAR; SCHED;
            STAGE(SB(0, 1), b2 + IMG);
            WAIT_V(6); BAR; MMA(1, 1, At, B1); BAR;
            LDB(B0, 1, 0); SCHED; LDA(At, 1, 0); STAGE(SA(0, 1), a2 + IMG);
            WAIT_L(8); BAR; WAIT_L(0); MMA(0, 0, At, B0); BAR; SCHED;
            LDB(B1, 1, 1); STAGE(SB(1, 0), b2 + KTB);
            BAR; WAIT_L(0); MMA(0, 1, At, B1); BAR;
            LDA(At, 1, 1); STAGE(SA(1, 0), a2 + KTB);
            BAR; WAIT_L(0); MMA(1, 0, At, B0); BAR; SCHED;
            STAGE(SB(1, 1), b2 + KTB + IMG);
            WAIT_V(6); BAR; MMA(1, 1, At, B1); BAR;
            a1 = a2 + KTB;
        }
        if (nt_done < 6) STAMP(6 + 4 * nt_done);
        unsigned tz = (unsigned)tid; asm volatile("" : "+v"(tz));
        const unsigned svo = tz * 16u;
        const __amdgpu_buffer_rsrc_t d_sl = mk_rsrc(P.ws[rank] + OFF_SLAB + (size_t)IT_S(cur) * SLAB, SLAB);
        unsigned* sflag = ctl + C_SLAB + IT_S(cur) * 8 + wid;
        if (SLAB16 && kind == 3) {
#pragma unroll
            for (int k = 0; k < 16; ++k) {
                const f32x4 v0 = acc[k >> 3][k & 1][(k >> 1) & 3][0], v1 = acc[k >> 3][k & 1][(k >> 1) & 3][1];
                u32x4 wv; wv.x = cvt_pk_bf16(v0[0], v0[1]); wv.y = cvt_pk_bf16(v0[2], v0[3]); wv.z = cvt_pk_bf16(v1[0], v1[1]); wv.w = cvt_pk_bf16(v1[2], v1[3]);
                if (bad) wv = (u32x4){0x7fc07fc0u, 0x7fc07fc0u, 0x7fc07fc0u, 0x7fc07fc0u};
                ST128(wv, d_sl, svo, (unsigned)k * 8192u, AUX_AGT);
            }
            WAIT_V(0);
            if (lane == 0) st_agt(sflag, call);
        } else if (kind == 1 || kind == 3) {
#pragma unroll
            for (int k = 0; k < 32; ++k) {
                u32x4 sv = __builtin_bit_cast(u32x4, acc[k >> 4][(k >> 1) & 1][(k >> 2) & 3][k & 1]);
                if (bad) sv = (u32x4){0x7fc00000u, 0x7fc00000u, 0x7fc00000u, 0x7fc00000u};
                ST128(sv, d_sl, svo, (unsigned)k * 8192u, AUX_AGT);
            }
            WAIT_V(0);
            if (lane == 0) st_agt(sflag, call);
        } else {
            const __amdgpu_buffer_rsrc_t d_out = mk_rsrc(P.out[rank], OUT_BYTES);
            const unsigned obase = (unsigned)(wr * 64 + (tz & 15u)) * OUT_PITCH + (unsigned)(wc * 32 + 8 * ((tz >> 4) & 3u)) * 2u;
            const unsigned sbase = (unsigned)(256 * IT_I(cur)) * OUT_PITCH + (unsigned)wpanel(IT_B(cur), rank) * 512u;
#define PUT(ai, m, bj, v0, v1) do { u32x4 wv; wv.x = cvt_pk_bf16(v0[0], v0[1]); wv.y = cvt_pk_bf16(v0[2], v0[3]); wv.z = cvt_pk_bf16(v1[0], v1[1]); wv.w = cvt_pk_bf16(v1[2], v1[3]); \
                if (bad) wv = (u32x4){0x7fc07fc0u, 0x7fc07fc0u, 0x7fc07fc0u, 0x7fc07fc0u}; \
                ST128(wv, d_out, obase + (bj) * 256u, sbase + (unsigned)((ai) * 128 + (m) * 16) * OUT_PITCH, AUX_SYS); } while (0)
            if (kind != 2) {
#pragma unroll
                for (int ai = 0; ai < 2; ++ai)
#pragma unroll
                    for (int m = 0; m < 4; ++m)
#pragma unroll
                        for (int bj = 0; bj < 2; ++bj) { const f32x4 v0 = acc[ai][bj][m][0], v1 = acc[ai][bj][m][1]; PUT(ai, m, bj, v0, v1); }
            } else {
                const u64 t0 = __builtin_amdgcn_s_memrealtime();
                for (;;) {
                    const unsigned f = (unsigned)RFL(ld_agt(sflag));
                    if (f >= call) break;
                    if (expired(ctl, call, t0, lane)) { bad = 1u; break; }
                    __builtin_amdgcn_s_sleep(4);
                }
#pragma unroll
                for (int g8 = 0; g8 < 4; ++g8) {
                    u32x4 pv[8];
#pragma unroll
                    for (int k = 0; k < 8; ++k) pv[k] = __builtin_amdgcn_raw_buffer_load_b128(d_sl, svo, (unsigned)(g8 * 8 + k) * 8192u, AUX_AGT);
#pragma unroll
                    for (int h = 0; h < 4; ++h) {
                        const int ai = g8 >> 1, m = (g8 & 1) * 2 + (h >> 1), bj = h & 1;
                        const f32x4 v0 = acc[ai][bj][m][0] + __builtin_bit_cast(f32x4, pv[2 * h]), v1 = acc[ai][bj][m][1] + __builtin_bit_cast(f32x4, pv[2 * h + 1]);
                        PUT(ai, m, bj, v0, v1);
                    }
                    SCHED;
                }
            }
#undef PUT
            if (IT_B(cur) < NSH) {
                WAIT_V(0);
                unsigned c = 0;
                if (lane == 0) c = __hip_atomic_fetch_add(mbox + 4, 1u, __ATOMIC_RELAXED, __HIP_MEMORY_SCOPE_WORKGROUP);
                if (((unsigned)RFL(c) & 7u) == 7u) signal_ranks(P, 0xffu & ~(1u << rank), C_YF + rank * 128 + IT_Y(cur), call, lane);
            }
        }
        if (nt_done < 6) STAMP(7 + 4 * nt_done);
        ++nt_done;
        if (nxt < 0) break;
        if (IT_K(nxt) == 4) {
            const __amdgpu_buffer_rsrc_t s_sl = mk_rsrc(P.ws[rank] + OFF_SLAB + (size_t)IT_S(nxt) * SLAB, SLAB);
            u32x4 pv[16];
#pragma unroll
            for (int k = 0; k < 16; ++k) pv[k] = __builtin_amdgcn_raw_buffer_load_b128(s_sl, svo, (unsigned)k * 8192u, AUX_AGT);
#pragma unroll
            for (int k = 0; k < 16; ++k) {
                acc[k >> 3][k & 1][(k >> 1) & 3][0] = __builtin_bit_cast(f32x4, (u32x4){pv[k].x << 16, pv[k].x & 0xffff0000u, pv[k].y << 16, pv[k].y & 0xffff0000u});
                acc[k >> 3][k & 1][(k >> 1) & 3][1] = __builtin_bit_cast(f32x4, (u32x4){pv[k].z << 16, pv[k].z & 0xffff0000u, pv[k].w << 16, pv[k].w & 0xffff0000u});
            }
#pragma unroll
            for (int k = 0; k < 32; k += 4)
                asm volatile("" : "+v"(acc[k >> 4][0][(k >> 2) & 3][0]), "+v"(acc[k >> 4][0][(k >> 2) & 3][1]), "+v"(acc[k >> 4][1][(k >> 2) & 3][0]), "+v"(acc[k >> 4][1][(k >> 2) & 3][1]));
        } else ZERO_ACC();
        bad = nbad;
        dyn = dyn || IT_K(nxt) != 3;
        cur = nxt; cA = nA; cB = nB;
    }
    WAIT_V(0);
    if (wr == 0) BAR;
#undef SA
#undef SB
#undef STAGE
#undef LDA
#undef LDB
#undef MMA
#undef ZERO_ACC
}

__global__ __launch_bounds__(512, 2) void fused_gemm_ar(Params P) {
    extern __shared__ __attribute__((aligned(16))) unsigned char shm[];
    LAS unsigned char* lds = (LAS unsigned char*)shm;
    const int b = blockIdx.x, rank = P.rank;
    unsigned* ctl = (unsigned*)P.ws[rank];
    const unsigned call = *(const CAS unsigned*)(u64)(ctl + C_FIN) / (unsigned)GRID + 1u;
    STAMP(0);
    if (b < NPULL) {
        puller(P, lds, b, call);
    } else {
        const int g = b - NPULL;
        stager(P, lds, g, call);
        STAMP(2);
        gemm_worker(P, lds, g, call);
    }
    __syncthreads();
    STAMP(1);
    STAMP(60 + (call & 1));
    if (threadIdx.x == 0) __hip_atomic_fetch_add(ctl + C_FIN, 1u, __ATOMIC_RELAXED, __HIP_MEMORY_SCOPE_AGENT);
}
}

extern "C" void kernel_launch(int rank, int nranks, ncclComm_t comm,
                              hipStream_t stream,
                              void* const* d_in, const int* in_sizes, int n_in,
                              void* d_out, int out_size,
                              const int* in_dtypes, int out_dtype,
                              void* const* d_out_all,
                              void* const* d_ws_all, size_t ws_bytes) {
    if (nranks != NR || (unsigned)rank >= (unsigned)NR || ws_bytes < WS_USED || n_in < 2 ||
        in_sizes[0] != MM * KS || in_sizes[1] != KS * NN || out_size != MM * NN) {
        fprintf(stderr, "kernel_launch: unexpected problem shape\n");
        return;
    }
    static unsigned attr_done = 0;
    int dev = 0; HIP_CHECK(hipGetDevice(&dev));
    if (!((attr_done >> (dev & 31)) & 1u)) {
        HIP_CHECK(hipFuncSetAttribute((const void*)fused_gemm_ar, hipFuncAttributeMaxDynamicSharedMemorySize, (int)LDS_BYTES));
        attr_done |= 1u << (dev & 31);
    }
    Params P;
    P.x = (const unsigned short*)d_in[0];
    P.w = (const unsigned short*)d_in[1];
    for (int p = 0; p < NR; ++p) { P.ws[p] = (char*)d_ws_all[p]; P.out[p] = (char*)d_out_all[p]; }
    P.rank = rank;
    hipLaunchKernelGGL(fused_gemm_ar, dim3(GRID), dim3(512), LDS_BYTES, stream, P);
    HIP_CHECK(hipGetLastError());
}
```

```cpp
#include <hip/hip_runtime.h>
#include <rccl/rccl.h>
#include <cstdio>
#include <cstdint>

#define HIP_CHECK(expr) do { hipError_t _e = (expr); if (_e != hipSuccess) { \
    fprintf(stderr, "HIP error %s at %s:%d\n", hipGetErrorString(_e), __FILE__, __LINE__); \
    return; } } while (0)

#define EREP 4
#define PUSH0 1
#define RTAIL 6
#define NPRE 48
#define PRIO1 40
#define PRIO2 112
#define DLY1 150
#define DLY2 300
#define SLAB16 1
#define PPG 6
#define WPL (PPG * 64 / 7)
#define KTICKS 100

namespace {
#define LAS __attribute__((address_space(3)))
#define GAS __attribute__((address_space(1)))
#define CAS __attribute__((address_space(4)))
typedef short bf16x8 __attribute__((ext_vector_type(8)));
typedef float f32x4 __attribute__((ext_vector_type(4)));
typedef unsigned u32x4 __attribute__((ext_vector_type(4)));
typedef unsigned long long u64;

constexpr int NR = 8;
constexpr int MM = 4096, NN = 16384, KS = 512;
constexpr unsigned IMG = 16384;
constexpr unsigned KTB = 2 * IMG;
constexpr unsigned CHUNK = 8 * KTB;
constexpr unsigned PANEL = NR * CHUNK;
constexpr int NXP = MM / 256, NWP = NN / 256;
constexpr int WPR = NWP / NR;
constexpr int NSH = WPR - EREP;
constexpr int NW = NSH + NR * EREP;
constexpr unsigned OFF_X = 1u << 20;
constexpr unsigned OFF_W = OFF_X + NXP * PANEL;
constexpr unsigned PAR_STRIDE = (NXP + NWP) * PANEL;
constexpr unsigned OFF_SLAB = OFF_X + 2 * PAR_STRIDE;
constexpr unsigned SLAB = 256 * 256 * 4;
constexpr unsigned WS_USED = OFF_SLAB + 512 * SLAB;
constexpr unsigned OUT_BYTES = (unsigned)MM * NN * 2;
constexpr unsigned OUT_PITCH = NN * 2;
constexpr int NCHUNK = NXP + NWP;
constexpr int XPUSH = PUSH0 ? 1 : 0;
constexpr int NSEQ = NXP + NW - XPUSH;
constexpr int PRE_COST = 150, MAXSLAB = 512;
constexpr int NYT = NXP * NSH;
constexpr int NIN_IT = NSEQ * 64;
constexpr int NTOT_IT = NIN_IT + NYT * 64;
constexpr int GRID = 256, NPULL = 8 * PPG, NWORK = GRID - NPULL, WPG = NWORK / 8;
constexpr int NJOB = NCHUNK * 4;
constexpr int C_FIN = 32, C_TMO = 64, C_SDONE = 96;
constexpr int C_XP = 128  ;
constexpr int C_GQ = 448, C_TQ = 768  ;
constexpr int C_Q = 5120  ;
constexpr int C_STG = 1024  ;
constexpr int C_YF = 8192  ;
constexpr int C_ARR = 12288  ;
constexpr int C_SLAB = 40960  ;
constexpr int C_ACNT = 16384  ;
constexpr unsigned LDS_BYTES = 8 * IMG + 256;
constexpr u64 DEADLINE = 40000000ull;
constexpr int AUX_SYS = 17;
constexpr int AUX_AGT = 16;
static_assert(RTAIL <= 16 && NSEQ <= 64 && NYT <= 128 && C_ACNT + 64 * 8 * 32 <= C_SLAB && (C_SLAB + 512 * 8) * 4 <= (512 << 10) && PPG >= 4, "layout");

struct Params {
    const unsigned short* x;
    const unsigned short* w;
    char* ws[NR];
    char* out[NR];
    int rank;
};

constexpr int pos_x(int i) { return 2 * i; }
constexpr int pos_w(int b) { return b < NXP ? 2 * b + 1 : b + NXP; }
constexpr int tile_pos(int i, int b) { return pos_x(i) > pos_w(b) ? pos_x(i) : pos_w(b); }
constexpr int NSEQ_POS = (NW >= NXP) ? NXP + NW : 2 * NXP;
#define IT_I(e) ((e) & 15)
#define IT_B(e) (((e) >> 4) & 63)
#define IT_Y(e) (((e) >> 10) & 127)
#define IT_K(e) (((e) >> 17) & 7)
#define IT_S(e) ((e) >> 20)
constexpr int MAXPRE = 4;
struct Tables {
    int tile[8][96];
    int ntile[8];
    int pre[8][WPG][MAXPRE + 1];
    int gtile[2 * 8 * 16];
    int ngt, nslab;
    int ytile[128];
    int seq[64];
};
constexpr Tables make_tables() {
    Tables T{};
    int ny = 0, ns = 0;
    for (int pos = 0; pos < NSEQ_POS; ++pos) {
        for (int i = XPUSH; i < NXP; ++i) if (pos_x(i) == pos) T.seq[ns++] = i;
        for (int b = 0; b < NW; ++b) if (pos_w(b) == pos) T.seq[ns++] = -1 - b;
        for (int b = 0; b < NW; ++b)
            for (int i = 0; i < NXP; ++i)
                if (tile_pos(i, b) == pos) {
                    const int q = (i & 3) + 4 * (b & 1);
                    T.tile[q][T.ntile[q]++] = i | (b << 4) | (b < NSH ? ny << 10 : 0);
                    if (b < NSH) T.ytile[ny++] = i | (b << 4);
                }
    }
    int gt[8 * 16] = {}; int ng = 0;
    for (int k = RTAIL; k >= 1; --k)
        for (int q = 0; q < 8; ++q)
            if (T.ntile[q] - RTAIL >= WPG) gt[ng++] = T.tile[q][T.ntile[q] - k];
    for (int q = 0; q < 8; ++q)
        if (T.ntile[q] - RTAIL >= WPG) T.ntile[q] -= RTAIL;
    constexpr int LAG = 8;
    for (int k = 0; k < ng + LAG; ++k) {
        if (k < ng) T.gtile[T.ngt++] = gt[k] | (1 << 17) | (k << 20);
        if (k >= LAG) T.gtile[T.ngt++] = gt[k - LAG] | (2 << 17) | ((k - LAG) << 20);
    }
    T.nslab = ng;
    for (int q = 0; q < 8; ++q) {
        int np[WPG] = {}, slack[WPG] = {};
        for (int w = 0; w < WPG; ++w) {
            for (int k = 0; k <= MAXPRE; ++k) T.pre[q][w][k] = -1;
            slack[w] = w < T.ntile[q] ? 60 + 44 * (tile_pos(IT_I(T.tile[q][w]), IT_B(T.tile[q][w])) + 1 - XPUSH) - 240 : 0;
        }
        for (int k = WPG; k < T.ntile[q] && k < WPG + NPRE; ++k) {
            int best = 0;
            for (int w = 1; w < WPG; ++w) if (slack[w] > slack[best]) best = w;
            if (slack[best] < PRE_COST || np[best] >= MAXPRE) break;
            T.pre[q][best][np[best]++] = T.tile[q][k] | (3 << 17) | (T.nslab << 20);
            T.tile[q][k] |= (4 << 17) | (T.nslab << 20);
            ++T.nslab; slack[best] -= PRE_COST;
            if (np[best] >= MAXPRE) slack[best] = 0;
        }
    }
    return T;
}
__constant__ const Tables TAB = make_tables();
constexpr int n_active() { const Tables T = make_tables(); int n = 0; for (int q = 0; q < 8; ++q) n += T.ntile[q] < WPG ? T.ntile[q] : WPG; return n; }
constexpr int NACTIVE = n_active();
static_assert(make_tables().nslab <= MAXSLAB, "slabs");

__device__ __forceinline__ int lds_byte(int r, int c) { const int st = (r >> 4) * 2 + (c >> 5), rr = r & 15, cc = c & 31, ob = rr * 64 + cc * 2; return st * 1024 + (ob ^ (((ob >> 9) & 1) << 5)); }
__device__ __forceinline__ void stage_rc(int b, int& R, int& C) { const int st = b / 1024, sb = b % 1024, swz = sb ^ (((sb >> 9) & 1) << 5); R = (st >> 1) * 16 + swz / 64; C = (st & 1) * 32 + (swz % 64) / 2; }
__device__ __forceinline__ int wpanel(int b, int r) { const int m = b - NSH; return b < NSH ? r * WPR + b : (m & 7) * WPR + NSH + (m >> 3); }
__device__ __forceinline__ int seq_chunk(int s, int r) { const int v = TAB.seq[s]; return v >= 0 ? v : NXP + wpanel(-1 - v, r); }
__device__ __forceinline__ unsigned chunk_off(int cid, int src, unsigned call) { return OFF_X + (call & 1u) * PAR_STRIDE + (unsigned)cid * PANEL + (unsigned)src * CHUNK; }

__device__ __forceinline__ __amdgpu_buffer_rsrc_t mk_rsrc(const void* p, unsigned bytes) {
    const u64 a = (u64)p;
    const unsigned lo = __builtin_amdgcn_readfirstlane((unsigned)a), hi = __builtin_amdgcn_readfirstlane((unsigned)(a >> 32));
    return __builtin_amdgcn_make_buffer_rsrc((void*)(((u64)hi << 32) | lo), 0, bytes, 0x00020000);
}
__device__ __forceinline__ unsigned ld_sys(const unsigned* p) { return __hip_atomic_load(p, __ATOMIC_RELAXED, __HIP_MEMORY_SCOPE_SYSTEM); }
__device__ __forceinline__ unsigned ld_agt(const unsigned* p) { return __hip_atomic_load(p, __ATOMIC_RELAXED, __HIP_MEMORY_SCOPE_AGENT); }
__device__ __forceinline__ void st_agt(unsigned* p, unsigned v) { __hip_atomic_store(p, v, __ATOMIC_RELAXED, __HIP_MEMORY_SCOPE_AGENT); }
#define RFL(x) __builtin_amdgcn_readfirstlane(x)
#define ST128(v, rsrc, voff, soff, aux) __builtin_amdgcn_raw_buffer_store_b128(v, rsrc, (voff) + (soff), 0, aux)
#define WAIT_V(n) asm volatile("s_waitcnt vmcnt(" #n ")" ::: "memory")
#define WAIT_L(n) asm volatile("s_waitcnt lgkmcnt(" #n ")" ::: "memory")
#define BAR __builtin_amdgcn_s_barrier()
#define SCHED __builtin_amdgcn_sched_barrier(0)
#define LANE0_ADD(rsrc, word, lane) (unsigned)__builtin_amdgcn_raw_ptr_buffer_atomic_add_i32(1, rsrc, (lane) == 0 ? (int)((word) * 4) : (int)0xfffffff0u, 0, 0)

__device__ __forceinline__ void signal_ranks(const Params& P, unsigned mask, int word, unsigned val, int lane) {
    const CAS Params* kp = (const CAS Params*)__builtin_amdgcn_kernarg_segment_ptr();
    asm volatile("" : "+s"(kp));
    u64 a = 0;
#pragma unroll
    for (int p = 0; p < NR; ++p) { u64 t = (u64)kp->ws[p]; asm volatile("" : "+s"(t)); a = (lane == p) ? t : a; }
    if (lane < NR && ((mask >> lane) & 1u)) __hip_atomic_store((GAS unsigned*)a + word, val, __ATOMIC_RELAXED, __HIP_MEMORY_SCOPE_SYSTEM);
}
__device__ __forceinline__ bool expired(unsigned* ctl, unsigned call, u64 t0, int lane) {
    const u64 dt = __builtin_amdgcn_s_memrealtime() - t0;
    if (dt < 100000ull) return false;
    const bool dead = dt > DEADLINE || (unsigned)RFL(ld_agt(ctl + C_TMO)) == call;
    if (dead && lane == 0) st_agt(ctl + C_TMO, call);
    return RFL((int)dead) != 0;
}

__device__ __forceinline__ void puller(const Params& P, LAS unsigned char* lds, int b, unsigned call) {
    const int rank = P.rank, lane = threadIdx.x & 63, wid = RFL((int)(threadIdx.x >> 6));
    unsigned* ctl = (unsigned*)P.ws[rank];
    const __amdgpu_buffer_rsrc_t d_ws = mk_rsrc(P.ws[rank], WS_USED), d_out = mk_rsrc(P.out[rank], OUT_BYTES);
    bool dead = false, xp_pend = false;
    LAS unsigned* xcnt = (LAS unsigned*)lds;
    const unsigned xp_peer = (unsigned)(rank + 1 + (b >> 2)) & 7u;
    if (threadIdx.x == 0) *xcnt = 0u;
    __syncthreads();
    if (wid < 4) {
        if (XPUSH && b < 28) {
            const __amdgpu_buffer_rsrc_t s_x = mk_rsrc(P.x, MM * KS * 2), p_ws = mk_rsrc(P.ws[xp_peer], WS_USED);
            const int im = (b & 3) * 4 + wid;
            u32x4 v[16];
#pragma unroll
            for (int q = 0; q < 16; ++q) {
                int R, C; stage_rc(q * 1024 + lane * 16, R, C);
                v[q] = __builtin_amdgcn_raw_buffer_load_b128(s_x, (unsigned)(R * KS + C) * 2u, (unsigned)(128 * (im & 1) * KS + 64 * (im >> 1)) * 2u, 0);
            }
#pragma unroll
            for (int q = 0; q < 16; ++q) ST128(v[q], p_ws, (unsigned)lane * 16u, chunk_off(0, rank, call) + (unsigned)im * IMG + q * 1024u, AUX_SYS);
            xp_pend = true;
        }
    }
    if (b == NPULL - 1 && threadIdx.x < 7) st_agt(ctl + C_Q + 256 * ((call + 1u) & 1u) + 32 * (int)threadIdx.x, 0u);
#define XP_DONE() do { if (XPUSH && xp_pend) { xp_pend = false; \
        if (lane == 0 && __hip_atomic_fetch_add(xcnt, 1u, __ATOMIC_RELAXED, __HIP_MEMORY_SCOPE_WORKGROUP) == 3u) \
            __hip_atomic_fetch_add((unsigned*)P.ws[xp_peer] + C_XP + 32 * rank, 1u, __ATOMIC_RELAXED, __HIP_MEMORY_SCOPE_SYSTEM); } } while (0)
    const int mycid = seq_chunk(lane < NSEQ ? lane : 0, rank);
    const unsigned ytab = (unsigned)TAB.ytile[lane] | ((unsigned)TAB.ytile[64 + lane] << 8);
    const int qw0 = C_Q + 256 * (int)(call & 1u);
    const int u = ((b >> 3) * 8 + wid) * 8 + (b & 7), link = u % 7;
    if (u >= 7 * WPL) { if (xp_pend) { WAIT_V(0); XP_DONE(); } return; }
    {
        const unsigned p = (unsigned)(rank + 1 + link) & 7u;
        const __amdgpu_buffer_rsrc_t s_ws = mk_rsrc(P.ws[p], WS_USED), s_out = mk_rsrc(P.out[p], OUT_BYTES);
        const int qword = qw0 + 32 * link;
        unsigned tk = LANE0_ADD(d_ws, qword, lane);
        unsigned nn = (unsigned)RFL(tk);
        u32x4 vl = {0u, 0u, 0u, 0u}; unsigned ol = 0;
        int pend = -1;
#define FLAG_IF_LAST(c, ps) do { if (((unsigned)RFL(c) & 63u) == 63u && lane == 0) st_agt(ctl + C_ARR + __builtin_amdgcn_readlane(mycid, ps) * 8 + p, call); } while (0)
#define FLUSH_IN() do { ST128(vl, d_ws, ol, 0u, AUX_AGT); WAIT_V(0); const unsigned _c = LANE0_ADD(d_ws, C_ACNT + (pend * 8 + (int)p) * 32, lane); FLAG_IF_LAST(_c, pend); pend = -1; } while (0)
        {
            const unsigned foff = (unsigned)(C_STG + p * 512 + mycid * 4) * 4u;
            u64 rq0 = 0, rq1 = 0, rq2 = 0, rq3 = 0;
            while (nn < (unsigned)NIN_IT) {
                const unsigned n = nn;
                const int s = (int)(n >> 6), qq = (int)(n >> 4) & 3;
                const bool rdy = (((qq == 0 ? rq0 : qq == 1 ? rq1 : qq == 2 ? rq2 : rq3) >> s) & 1) != 0;
                if (!rdy && !dead) {
                    const u64 t0 = __builtin_amdgcn_s_memrealtime();
                    for (;;) {
                        const u32x4 f = __builtin_amdgcn_raw_buffer_load_b128(d_ws, foff, 0, AUX_SYS);
                        rq0 = __ballot(f.x >= call); rq1 = __ballot(f.y >= call); rq2 = __ballot(f.z >= call); rq3 = __ballot(f.w >= call);
                        if (((qq == 0 ? rq0 : qq == 1 ? rq1 : qq == 2 ? rq2 : rq3) >> s) & 1) break;
                        if (xp_pend) { WAIT_V(0); XP_DONE(); }
                        if (pend >= 0) FLUSH_IN();
                        if (expired(ctl, call, t0, lane)) { dead = true; break; }
                        __builtin_amdgcn_s_sleep(1);
                    }
                }
                const int cid = __builtin_amdgcn_readlane(mycid, s);
                const unsigned soff = chunk_off(cid, (int)p, call) + (n & 63u) * 4096u, voff = (unsigned)lane * 16u;
                const unsigned so = dead ? 0xfffff000u : soff;
                tk = LANE0_ADD(d_ws, qword, lane);
                const u32x4 v0 = __builtin_amdgcn_raw_buffer_load_b128(s_ws, voff, so, AUX_SYS);
                ST128(vl, d_ws, pend >= 0 ? ol : 0xfffffff0u, 0u, AUX_AGT);
                WAIT_V(1);
                const u32x4 v1 = __builtin_amdgcn_raw_buffer_load_b128(s_ws, voff + 1024u, so, AUX_SYS);
                ST128(v0, d_ws, voff, soff, AUX_AGT);
                nn = (unsigned)RFL(tk);
                WAIT_V(1);
                XP_DONE();
                const unsigned c = LANE0_ADD(d_ws, pend >= 0 ? C_ACNT + (pend * 8 + (int)p) * 32 : 0x3ffffff0, lane);
                const u32x4 v2 = __builtin_amdgcn_raw_buffer_load_b128(s_ws, voff + 2048u, so, AUX_SYS);
                ST128(v1, d_ws, voff, soff + 1024u, AUX_AGT);
                WAIT_V(1);
                if (pend >= 0) FLAG_IF_LAST(c, pend);
                const u32x4 v3 = __builtin_amdgcn_raw_buffer_load_b128(s_ws, voff + 3072u, so, AUX_SYS);
                ST128(v2, d_ws, voff, soff + 2048u, AUX_AGT);
                vl = v3; ol = voff + soff + 3072u; pend = s;
                asm volatile("" : "+v"(vl));
            }
            if (pend >= 0) FLUSH_IN();
            if (xp_pend) { WAIT_V(0); XP_DONE(); }
        }
        {
            const unsigned* fl = ctl + C_YF + p * 128 + lane;
            const unsigned voff = (unsigned)(lane >> 5) * OUT_PITCH + p * (WPR * 512u) + (unsigned)(lane & 31) * 16u;
            u64 r0 = 0, r1 = 0;
            ol = 0xfffffff0u;
            while (nn < (unsigned)NTOT_IT) {
                const unsigned n = nn - (unsigned)NIN_IT;
                const int o = (int)(n >> 6);
                if (!dead && !(((o < 64 ? r0 : r1) >> (o & 63)) & 1)) {
                    const u64 t0 = __builtin_amdgcn_s_memrealtime();
                    for (;;) {
                        const unsigned f0 = ld_sys(fl), f1 = ld_sys(fl + 64);
                        r0 = __ballot(f0 >= call); r1 = __ballot(f1 >= call);
                        if (((o < 64 ? r0 : r1) >> (o & 63)) & 1) break;
                        if (expired(ctl, call, t0, lane)) { dead = true; break; }
                        __builtin_amdgcn_s_sleep(1);
                    }
                }
                const unsigned e = ((unsigned)__builtin_amdgcn_readlane((int)ytab, o & 63) >> ((o >> 6) * 8)) & 255u;
                const unsigned soff = ((e & 15u) * 256u + (n & 63u) * 4u) * OUT_PITCH + (e >> 4) * 512u;
                const int o2 = o + 3 < NYT ? o + 3 : NYT - 1;
                const bool refresh = !(((o2 < 64 ? r0 : r1) >> (o2 & 63)) & 1);
                unsigned f0 = 0, f1 = 0;
                if (refresh) { f0 = ld_sys(fl); f1 = ld_sys(fl + 64); }
                tk = LANE0_ADD(d_ws, qword, lane);
                u32x4 v0 = __builtin_amdgcn_raw_buffer_load_b128(s_out, voff, soff, AUX_SYS);
                ST128(vl, d_out, ol, 0u, AUX_AGT);
                WAIT_V(1);
                u32x4 v1 = __builtin_amdgcn_raw_buffer_load_b128(s_out, voff, soff + 2 * OUT_PITCH, AUX_SYS);
                if (dead) v0 = (u32x4){0x7fc07fc0u, 0x7fc07fc0u, 0x7fc07fc0u, 0x7fc07fc0u};
                ST128(v0, d_out, voff, soff, AUX_AGT);
                nn = (unsigned)RFL(tk);
                if (refresh) { r0 = __ballot(f0 >= call); r1 = __ballot(f1 >= call); }
                if (dead) v1 = (u32x4){0x7fc07fc0u, 0x7fc07fc0u, 0x7fc07fc0u, 0x7fc07fc0u};
                vl = v1; ol = voff + soff + 2 * OUT_PITCH;
                asm volatile("" : "+v"(vl));
            }
            ST128(vl, d_out, ol, 0u, AUX_AGT);
        }
    }
#undef FLUSH_IN
#undef FLAG_IF_LAST
#undef XP_DONE
}

__device__ __forceinline__ void job_of(int idx, int& cid, int& q) {
    q = idx & 3;
    int u = idx >> 2;
    if (u < 9 * NSH) { const int g = u / 9, r = u % 9; cid = r < 8 ? NXP + r * WPR + g : (g == 0 ? XPUSH : g == 1 ? 1 - XPUSH : g); return; }
    u -= 9 * NSH;
    constexpr int NXR = NXP - NSH, NPAIR = NXR < 8 * EREP ? NXR : 8 * EREP;
    if (u < 2 * NPAIR) { cid = (u & 1) ? NXP + wpanel(NSH + (u >> 1), 0) : NSH + (u >> 1); return; }
    u -= 2 * NPAIR;
    cid = NXR > NPAIR ? NSH + NPAIR + u : NXP + wpanel(NSH + NPAIR + u, 0);
}
__device__ __forceinline__ void stager(const Params& P, LAS unsigned char* lds, int g, unsigned call) {
    const int rank = P.rank, tid = threadIdx.x, lane = tid & 63;
    const __amdgpu_buffer_rsrc_t d_ws = mk_rsrc(P.ws[rank], WS_USED);
    const __amdgpu_buffer_rsrc_t s_x = mk_rsrc(P.x, MM * KS * 2), s_w = mk_rsrc(P.w, KS * NN * 2);
    if (g >= PRIO1) {
        const u64 t0 = __builtin_amdgcn_s_memrealtime(), dly = g >= PRIO2 ? DLY2 : DLY1;
        while (__builtin_amdgcn_s_memrealtime() - t0 < dly) __builtin_amdgcn_s_sleep(4);
    }
    for (int idx = g; idx < NJOB; idx += NWORK) {
        int cid, q; job_of(idx, cid, q);
        const unsigned dst = chunk_off(cid, rank, call) + (unsigned)q * (2 * KTB) + (unsigned)tid * 16u;
        if (cid < NXP) {
            u32x4 v[8];
#pragma unroll
            for (int u = 0; u < 8; ++u) {
                int R, C; stage_rc((u & 1) * 8192 + tid * 16, R, C);
                v[u] = __builtin_amdgcn_raw_buffer_load_b128(s_x, (unsigned)(R * KS + C) * 2u, (unsigned)((256 * cid + 128 * ((u >> 1) & 1)) * KS + 128 * q + 64 * (u >> 2)) * 2u, 0);
            }
#pragma unroll
            for (int u = 0; u < 8; ++u) ST128(v[u], d_ws, dst + u * 8192u, 0, AUX_SYS);
        } else {
            const int jg = cid - NXP, ng = tid & 31, kg = tid >> 5;
            u32x4 v[8];
#pragma unroll
            for (int r = 0; r < 8; ++r) v[r] = __builtin_amdgcn_raw_buffer_load_b128(s_w, (unsigned)(8 * kg * NN + 8 * ng) * 2u, (unsigned)((128 * q + r) * NN + 256 * jg) * 2u, 0);
            const int lbase = (kg >> 3) * (int)KTB + (ng >> 4) * (int)IMG;
#pragma unroll
            for (int jj = 0; jj < 8; ++jj) {
                u32x4 o;
#pragma unroll
                for (int d = 0; d < 4; ++d)
                    o[d] = (jj & 1) ? __builtin_amdgcn_perm(v[2 * d + 1][jj >> 1], v[2 * d][jj >> 1], 0x07060302u)
                                    : __builtin_amdgcn_perm(v[2 * d + 1][jj >> 1], v[2 * d][jj >> 1], 0x05040100u);
                const int R = 32 * ((ng & 15) >> 2) + 16 * (jj >> 2) + 4 * (ng & 3) + (jj & 3);
                *(LAS u32x4*)(lds + lbase + lds_byte(R, (kg & 7) * 8)) = o;
            }
            __syncthreads();
#pragma unroll
            for (int u = 0; u < 8; ++u) v[u] = *(const LAS u32x4*)(lds + u * 8192 + tid * 16);
#pragma unroll
            for (int u = 0; u < 8; ++u) ST128(v[u], d_ws, dst + u * 8192u, 0, AUX_SYS);
        }
        WAIT_V(0);
        __syncthreads();
        const unsigned mask = cid < XPUSH ? (1u << rank) : (cid < NXP || ((cid - NXP) & 7) >= NSH) ? 0xffu : (1u << ((cid - NXP) >> 3));
        if (tid < 64) signal_ranks(P, mask, C_STG + rank * 512 + cid * 4 + q, call, lane);
    }
    if (tid == 0) __hip_atomic_fetch_add((unsigned*)P.ws[rank] + C_SDONE, 1u, __ATOMIC_RELAXED, __HIP_MEMORY_SCOPE_AGENT);
}

__device__ __forceinline__ unsigned cvt_pk_bf16(float lo, float hi) { unsigned r; asm volatile("v_cvt_pk_bf16_f32 %0, %1, %2" : "=v"(r) : "v"(lo), "v"(hi)); return r; }

__device__ __forceinline__ unsigned wait_inputs(unsigned* ctl, unsigned call, int rank, int lane, bool arr, int i, int jg, bool sd, int slab) {
    const bool xp = i < XPUSH;
    const unsigned* w0 = lane >= 24 ? ctl + C_SLAB + (slab < 0 ? 0 : slab) * 8 + (lane & 7)
                       : lane == 16 ? ctl + C_SDONE : lane == 17 ? ctl + C_TMO
                       : (xp && lane < 8) ? ctl + C_XP + 32 * lane : ctl + C_ARR + ((lane & 8) ? NXP + jg : i) * 8 + (lane & 7);
    const unsigned tgt = lane == 16 ? (unsigned)NWORK * call : (xp && lane < 8) ? 4u * call : call;
    const bool act = lane < 16 ? (arr && (lane & 7) != rank) : lane == 16 ? sd : (lane >= 24 && lane < 32 && slab >= 0);
    const u64 t0 = __builtin_amdgcn_s_memrealtime();
    unsigned bad = 0;
    for (;;) {
        const unsigned f = (act || lane == 17) ? ld_sys(w0) : tgt;
        if ((unsigned)__builtin_amdgcn_readlane((int)f, 17) == call) { bad = 1u; break; }
        if (!__ballot(act && (int)(f - tgt) < 0)) break;
        if (expired(ctl, call, t0, lane)) { bad = 1u; break; }
        __builtin_amdgcn_s_sleep(8);
    }
    if (arr && xp) { __builtin_amdgcn_fence(__ATOMIC_ACQUIRE, ""); WAIT_V(0); }
    return bad;
}

__device__ __forceinline__ void gemm_worker(const Params& P, LAS unsigned char* lds, int g, unsigned call) {
    const int rank = P.rank, grp = g & 7;
    const int ntq = TAB.ntile[grp];
    if ((g >> 3) >= ntq) return;
    const int tid = threadIdx.x, wid = RFL(tid >> 6), lane = tid & 63, wr = wid >> 2, wc = wid & 3, fr = lane & 15, fq = lane >> 4;
    unsigned* ctl = (unsigned*)P.ws[rank];
    LAS unsigned* mbox = (LAS unsigned*)(lds + 8 * IMG);
    const char* wsb = P.ws[rank] + (size_t)((call & 1u) * PAR_STRIDE);
    const unsigned voff0 = (unsigned)tid * 16u, voff1 = voff0 + 8192u;
    const unsigned ldsw = (unsigned)wid * 1024u;
    const int aoff = lds_byte(wr * 64 + fr, fq * 8), boff = lds_byte(wc * 32 + fr, fq * 8);
    const int own8 = rank * 8;
    const unsigned ownoff = (unsigned)own8 * KTB;
#define SA(b, h) (((b) * 2 + (h)) * IMG)
#define SB(b, h) ((4 + (b) * 2 + (h)) * IMG)
    const __amdgpu_buffer_rsrc_t s_pan = mk_rsrc(wsb, OFF_X + PAR_STRIDE);
#define STAGE(bufoff, gbase) do { \
        __builtin_amdgcn_raw_ptr_buffer_load_lds(s_pan, (LAS void*)(lds + (bufoff) + ldsw), 16, voff0, (gbase), 0, 0); \
        __builtin_amdgcn_raw_ptr_buffer_load_lds(s_pan, (LAS void*)(lds + (bufoff) + ldsw + 8192), 16, voff1, (gbase), 0, 0); } while (0)
#define LDA(dst, b, h) do { _Pragma("unroll") for (int m = 0; m < 4; ++m) _Pragma("unroll") for (int k = 0; k < 2; ++k) dst[m][k] = *(const LAS bf16x8*)(lds + SA(b, h) + aoff + m * 2048 + k * 1024); } while (0)
#define LDB(dst, b, h) do { _Pragma("unroll") for (int n = 0; n < 2; ++n) _Pragma("unroll") for (int k = 0; k < 2; ++k) dst[n][k] = *(const LAS bf16x8*)(lds + SB(b, h) + boff + n * 2048 + k * 1024); } while (0)
#define MMA(ai, bj, At, Bt) do { __builtin_amdgcn_s_setprio(1); _Pragma("unroll") for (int m = 0; m < 4; ++m) _Pragma("unroll") for (int n = 0; n < 2; ++n) _Pragma("unroll") for (int k = 0; k < 2; ++k) \
        acc[ai][bj][m][n] = __builtin_amdgcn_mfma_f32_16x16x32_bf16(Bt[n][k], At[m][k], acc[ai][bj][m][n], 0, 0, 0); __builtin_amdgcn_s_setprio(0); } while (0)
#define ZERO_ACC() do { _Pragma("unroll") for (int a = 0; a < 2; ++a) _Pragma("unroll") for (int b = 0; b < 2; ++b) _Pragma("unroll") for (int m = 0; m < 4; ++m) _Pragma("unroll") for (int n = 0; n < 2; ++n) \
        acc[a][b][m][n] = (f32x4){0.f, 0.f, 0.f, 0.f}; } while (0)
    f32x4 acc[2][2][4][2];
    ZERO_ACC();
    bf16x8 At[4][2], B0[2][2], B1[2][2];
    const int slot = g >> 3;
    int sidx = 0;
    int cur = TAB.pre[grp][slot][0];
    if (cur < 0) cur = TAB.tile[grp][slot];
    bool dyn = IT_K(cur) != 3;
    unsigned cA = OFF_X + (unsigned)IT_I(cur) * PANEL, cB = OFF_W + (unsigned)wpanel(IT_B(cur), rank) * PANEL;
    if (wid == 0) {
        if (lane == 0) mbox[4] = 0u;
        const u64 t0 = __builtin_amdgcn_s_memrealtime();
        const unsigned* w0 = ctl + C_STG + rank * 512 + ((lane & 4) ? NXP + wpanel(IT_B(cur), rank) : IT_I(cur)) * 4 + (lane & 3);
        for (;;) {
            const unsigned f = lane < 8 ? ld_agt(w0) : call;
            if (__all(f >= call)) break;
            if (expired(ctl, call, t0, lane)) break;
            __builtin_amdgcn_s_sleep(4);
        }
        __builtin_amdgcn_fence(__ATOMIC_ACQUIRE, "agent");
        WAIT_V(0); WAIT_L(0);
    }
    asm volatile("" ::: "memory"); BAR; asm volatile("" ::: "memory");
    STAGE(SB(0, 0), cB + ownoff); STAGE(SA(0, 0), cA + ownoff); STAGE(SB(0, 1), cB + ownoff + IMG); STAGE(SA(0, 1), cA + ownoff + IMG);
    if (wr == 1) BAR;
    WAIT_V(4); BAR;
    STAGE(SB(1, 0), cB + ownoff + KTB); STAGE(SA(1, 0), cA + ownoff + KTB); STAGE(SB(1, 1), cB + ownoff + KTB + IMG);
    WAIT_V(6); BAR;
    unsigned a1 = cA + ownoff + KTB;
    int nt_done = 0, rho = 0;
    unsigned bad = 0;
    bool use_gq = false, drew = false;
    for (;;) {
        const int kind = IT_K(cur);
        const bool half = kind == 1 || kind == 2;
        const int nown = kind == 4 ? 0 : half ? 4 : 8, nkt = nown + (kind == 3 ? 0 : half ? 28 : 56);
        const int th = (kind == 0 || kind == 3) ? 6 : 2, rlo = kind == 2 ? 28 : 0, rhi = kind == 1 ? 28 : 56;
        const unsigned own0 = ownoff + (kind == 2 ? 4u * KTB : 0u);
        int nxt = -1; unsigned nbad = 0;
        unsigned nA = cA, nB = cB;
        for (int t = 0; t < nkt; t += 2) {
            const bool hookA = t == th, hookB = t == nkt - 2 && nxt >= 0 && IT_K(nxt) == 4;
            if (hookA && wid == 0) {
                const __amdgpu_buffer_rsrc_t d_ws = mk_rsrc(ctl, 1u << 20);
                const int qw = use_gq ? C_GQ : C_TQ + 32 * grp;
                unsigned tk = 0;
                if (dyn && drew) tk = LANE0_ADD(d_ws, qw, lane);
                unsigned tm = 0;
                if (kind != 4) tm = wait_inputs(ctl, call, rank, lane, kind != 3, IT_I(cur), wpanel(IT_B(cur), rank), !drew, -1);
                int e = -1;
                if (!dyn) {
                    e = TAB.pre[grp][slot][++sidx];
                    if (e < 0) e = TAB.tile[grp][slot];
                } else {
                    if (!drew) { tk = LANE0_ADD(d_ws, qw, lane); drew = true; }
                    if (!use_gq) {
                        const unsigned nx = (unsigned)WPG + (unsigned)RFL(tk) - (call - 1u) * (unsigned)ntq;
                        if (nx < (unsigned)ntq) e = TAB.tile[grp][nx];
                        else { use_gq = true; tk = LANE0_ADD(d_ws, C_GQ, lane); }
                    }
                    if (use_gq) {
                        const unsigned nx = (unsigned)RFL(tk) - (call - 1u) * (unsigned)(TAB.ngt + NACTIVE);
                        if (nx < (unsigned)TAB.ngt) e = TAB.gtile[nx];
                    }
                }
                asm volatile("" :: "v"(tk));
                if (lane == 0) {
                    const unsigned ph = (unsigned)((__builtin_amdgcn_s_memrealtime() / (2 * KTICKS)) % 28ull);
                    mbox[0] = half ? (unsigned)rlo + (ph % 14u) * 2u : ph * 2u;
                    mbox[1] = (unsigned)e;
                    mbox[2] = tm;
                }
            }
            if (hookB && wid == 0) {
                const unsigned tm = wait_inputs(ctl, call, rank, lane, true, IT_I(nxt), wpanel(IT_B(nxt), rank), false, IT_S(nxt));
                if (lane == 0) {
                    mbox[0] = (unsigned)((__builtin_amdgcn_s_memrealtime() / (2 * KTICKS)) % 28ull) * 2u;
                    mbox[2] = tm;
                }
            }
            LDB(B0, 0, 0); SCHED; LDA(At, 0, 0); STAGE(SA(1, 1), a1 + IMG);
            WAIT_L(8); BAR;
            u32x4 mb = {0u, 0u, 0u, 0u};
            if (hookA || hookB) mb = *(volatile LAS u32x4*)mbox;
            WAIT_L(0); MMA(0, 0, At, B0); BAR; SCHED;
            if (hookA) {
                if (kind != 4) { rho = RFL((int)mb.x); bad = (unsigned)RFL((int)mb.z); }
                nxt = RFL((int)mb.y);
                if (nxt >= 0) { nA = OFF_X + (unsigned)IT_I(nxt) * PANEL; nB = OFF_W + (unsigned)wpanel(IT_B(nxt), rank) * PANEL; }
            }
            unsigned a2, b2;
            if (t + 2 < nown) { a2 = cA + own0 + (unsigned)(t + 2) * KTB; b2 = cB + own0 + (unsigned)(t + 2) * KTB; }
            else if (t + 2 < nkt) { const unsigned ko = (unsigned)(rho + (rho >= own8 ? 8 : 0)) * KTB; a2 = cA + ko; b2 = cB + ko; rho = rho + 2 == rhi ? rlo : rho + 2; }
            else if (hookB) {
                rho = RFL((int)mb.x); nbad = (unsigned)RFL((int)mb.z);
                const unsigned ko = (unsigned)(rho + (rho >= own8 ? 8 : 0)) * KTB; a2 = nA + ko; b2 = nB + ko; rho = rho == 54 ? 0 : rho + 2;
            } else { const unsigned no = ownoff + ((nxt >= 0 && IT_K(nxt) == 2) ? 4u * KTB : 0u); a2 = nA + no; b2 = nB + no; }
            LDB(B1, 0, 1); STAGE(SB(0, 0), b2);
            BAR; WAIT_L(0); MMA(0, 1, At, B1); BAR;
            LDA(At, 0, 1); STAGE(SA(0, 0), a2);
            BAR; WAIT_L(0); MMA(1, 0, At, B0); BAR; SCHED;
            STAGE(SB(0, 1), b2 + IMG);
            WAIT_V(6); BAR; MMA(1, 1, At, B1); BAR;
            LDB(B0, 1, 0); SCHED; LDA(At, 1, 0); STAGE(SA(0, 1), a2 + IMG);
            WAIT_L(8); BAR; WAIT_L(0); MMA(0, 0, At, B0); BAR; SCHED;
            LDB(B1, 1, 1); STAGE(SB(1, 0), b2 + KTB);
            BAR; WAIT_L(0); MMA(0, 1, At, B1); BAR;
            LDA(At, 1, 1); STAGE(SA(1, 0), a2 + KTB);
            BAR; WAIT_L(0); MMA(1, 0, At, B0); BAR; SCHED;
            STAGE(SB(1, 1), b2 + KTB + IMG);
            WAIT_V(6); BAR; MMA(1, 1, At, B1); BAR;
            a1 = a2 + KTB;
        }
        unsigned tz = (unsigned)tid; asm volatile("" : "+v"(tz));
        const unsigned svo = tz * 16u;
        const __amdgpu_buffer_rsrc_t d_sl = mk_rsrc(P.ws[rank] + OFF_SLAB + (size_t)IT_S(cur) * SLAB, SLAB);
        unsigned* sflag = ctl + C_SLAB + IT_S(cur) * 8 + wid;
        if (SLAB16 && kind == 3) {
#pragma unroll
            for (int k = 0; k < 16; ++k) {
                const f32x4 v0 = acc[k >> 3][k & 1][(k >> 1) & 3][0], v1 = acc[k >> 3][k & 1][(k >> 1) & 3][1];
                u32x4 wv; wv.x = cvt_pk_bf16(v0[0], v0[1]); wv.y = cvt_pk_bf16(v0[2], v0[3]); wv.z = cvt_pk_bf16(v1[0], v1[1]); wv.w = cvt_pk_bf16(v1[2], v1[3]);
                if (bad) wv = (u32x4){0x7fc07fc0u, 0x7fc07fc0u, 0x7fc07fc0u, 0x7fc07fc0u};
                ST128(wv, d_sl, svo, (unsigned)k * 8192u, AUX_AGT);
            }
            WAIT_V(0);
            if (lane == 0) st_agt(sflag, call);
        } else if (kind == 1 || kind == 3) {
#pragma unroll
            for (int k = 0; k < 32; ++k) {
                u32x4 sv = __builtin_bit_cast(u32x4, acc[k >> 4][(k >> 1) & 1][(k >> 2) & 3][k & 1]);
                if (bad) sv = (u32x4){0x7fc00000u, 0x7fc00000u, 0x7fc00000u, 0x7fc00000u};
                ST128(sv, d_sl, svo, (unsigned)k * 8192u, AUX_AGT);
            }
            WAIT_V(0);
            if (lane == 0) st_agt(sflag, call);
        } else {
            const __amdgpu_buffer_rsrc_t d_out = mk_rsrc(P.out[rank], OUT_BYTES);
            const unsigned obase = (unsigned)(wr * 64 + (tz & 15u)) * OUT_PITCH + (unsigned)(wc * 32 + 8 * ((tz >> 4) & 3u)) * 2u;
            const unsigned sbase = (unsigned)(256 * IT_I(cur)) * OUT_PITCH + (unsigned)wpanel(IT_B(cur), rank) * 512u;
#define PUT(ai, m, bj, v0, v1) do { u32x4 wv; wv.x = cvt_pk_bf16(v0[0], v0[1]); wv.y = cvt_pk_bf16(v0[2], v0[3]); wv.z = cvt_pk_bf16(v1[0], v1[1]); wv.w = cvt_pk_bf16(v1[2], v1[3]); \
                if (bad) wv = (u32x4){0x7fc07fc0u, 0x7fc07fc0u, 0x7fc07fc0u, 0x7fc07fc0u}; \
                ST128(wv, d_out, obase + (bj) * 256u, sbase + (unsigned)((ai) * 128 + (m) * 16) * OUT_PITCH, AUX_SYS); } while (0)
            if (kind != 2) {
#pragma unroll
                for (int ai = 0; ai < 2; ++ai)
#pragma unroll
                    for (int m = 0; m < 4; ++m)
#pragma unroll
                        for (int bj = 0; bj < 2; ++bj) { const f32x4 v0 = acc[ai][bj][m][0], v1 = acc[ai][bj][m][1]; PUT(ai, m, bj, v0, v1); }
            } else {
                const u64 t0 = __builtin_amdgcn_s_memrealtime();
                for (;;) {
                    const unsigned f = (unsigned)RFL(ld_agt(sflag));
                    if (f >= call) break;
                    if (expired(ctl, call, t0, lane)) { bad = 1u; break; }
                    __builtin_amdgcn_s_sleep(4);
                }
#pragma unroll
                for (int g8 = 0; g8 < 4; ++g8) {
                    u32x4 pv[8];
#pragma unroll
                    for (int k = 0; k < 8; ++k) pv[k] = __builtin_amdgcn_raw_buffer_load_b128(d_sl, svo, (unsigned)(g8 * 8 + k) * 8192u, AUX_AGT);
#pragma unroll
                    for (int h = 0; h < 4; ++h) {
                        const int ai = g8 >> 1, m = (g8 & 1) * 2 + (h >> 1), bj = h & 1;
                        const f32x4 v0 = acc[ai][bj][m][0] + __builtin_bit_cast(f32x4, pv[2 * h]), v1 = acc[ai][bj][m][1] + __builtin_bit_cast(f32x4, pv[2 * h + 1]);
                        PUT(ai, m, bj, v0, v1);
                    }
                    SCHED;
                }
            }
#undef PUT
            if (IT_B(cur) < NSH) {
                WAIT_V(0);
                unsigned c = 0;
                if (lane == 0) c = __hip_atomic_fetch_add(mbox + 4, 1u, __ATOMIC_RELAXED, __HIP_MEMORY_SCOPE_WORKGROUP);
                if (((unsigned)RFL(c) & 7u) == 7u) signal_ranks(P, 0xffu & ~(1u << rank), C_YF + rank * 128 + IT_Y(cur), call, lane);
            }
        }
        ++nt_done;
        if (nxt < 0) break;
        if (IT_K(nxt) == 4) {
            const __amdgpu_buffer_rsrc_t s_sl = mk_rsrc(P.ws[rank] + OFF_SLAB + (size_t)IT_S(nxt) * SLAB, SLAB);
            u32x4 pv[16];
#pragma unroll
            for (int k = 0; k < 16; ++k) pv[k] = __builtin_amdgcn_raw_buffer_load_b128(s_sl, svo, (unsigned)k * 8192u, AUX_AGT);
#pragma unroll
            for (int k = 0; k < 16; ++k) {
                acc[k >> 3][k & 1][(k >> 1) & 3][0] = __builtin_bit_cast(f32x4, (u32x4){pv[k].x << 16, pv[k].x & 0xffff0000u, pv[k].y << 16, pv[k].y & 0xffff0000u});
                acc[k >> 3][k & 1][(k >> 1) & 3][1] = __builtin_bit_cast(f32x4, (u32x4){pv[k].z << 16, pv[k].z & 0xffff0000u, pv[k].w << 16, pv[k].w & 0xffff0000u});
            }
#pragma unroll
            for (int k = 0; k < 32; k += 4)
                asm volatile("" : "+v"(acc[k >> 4][0][(k >> 2) & 3][0]), "+v"(acc[k >> 4][0][(k >> 2) & 3][1]), "+v"(acc[k >> 4][1][(k >> 2) & 3][0]), "+v"(acc[k >> 4][1][(k >> 2) & 3][1]));
        } else ZERO_ACC();
        bad = nbad;
        dyn = dyn || IT_K(nxt) != 3;
        cur = nxt; cA = nA; cB = nB;
    }
    WAIT_V(0);
    if (wr == 0) BAR;
#undef SA
#undef SB
#undef STAGE
#undef LDA
#undef LDB
#undef MMA
#undef ZERO_ACC
}

__global__ __launch_bounds__(512, 2) void fused_gemm_ar(Params P) {
    extern __shared__ __attribute__((aligned(16))) unsigned char shm[];
    LAS unsigned char* lds = (LAS unsigned char*)shm;
    const int b = blockIdx.x, rank = P.rank;
    unsigned* ctl = (unsigned*)P.ws[rank];
    const unsigned call = *(const CAS unsigned*)(u64)(ctl + C_FIN) / (unsigned)GRID + 1u;
    if (b < NPULL) {
        puller(P, lds, b, call);
    } else {
        const int g = b - NPULL;
        stager(P, lds, g, call);
        gemm_worker(P, lds, g, call);
    }
    __syncthreads();
    if (threadIdx.x == 0) __hip_atomic_fetch_add(ctl + C_FIN, 1u, __ATOMIC_RELAXED, __HIP_MEMORY_SCOPE_AGENT);
}
}

extern "C" void kernel_launch(int rank, int nranks, ncclComm_t comm,
                              hipStream_t stream,
                              void* const* d_in, const int* in_sizes, int n_in,
                              void* d_out, int out_size,
                              const int* in_dtypes, int out_dtype,
                              void* const* d_out_all,
                              void* const* d_ws_all, size_t ws_bytes) {
    if (nranks != NR || (unsigned)rank >= (unsigned)NR || ws_bytes < WS_USED || n_in < 2 ||
        in_sizes[0] != MM * KS || in_sizes[1] != KS * NN || out_size != MM * NN) {
        fprintf(stderr, "kernel_launch: unexpected problem shape\n");
        return;
    }
    static unsigned attr_done = 0;
    int dev = 0; HIP_CHECK(hipGetDevice(&dev));
    if (!((attr_done >> (dev & 31)) & 1u)) {
        HIP_CHECK(hipFuncSetAttribute((const void*)fused_gemm_ar, hipFuncAttributeMaxDynamicSharedMemorySize, (int)LDS_BYTES));
        attr_done |= 1u << (dev & 31);
    }
    Params P;
    P.x = (const unsigned short*)d_in[0];
    P.w = (const unsigned short*)d_in[1];
    for (int p = 0; p < NR; ++p) { P.ws[p] = (char*)d_ws_all[p]; P.out[p] = (char*)d_out_all[p]; }
    P.rank = rank;
    hipLaunchKernelGGL(fused_gemm_ar, dim3(GRID), dim3(512), LDS_BYTES, stream, P);
    HIP_CHECK(hipGetLastError());
}
```
